```python
import numpy as np

kernel_name = "gpu_a2a_gemm_m8192_k8192_n8192_f32_gelu"


ABSMAX_RTOL = 0.001

COMPUTE_DTYPE = "float32"
EPILOGUE = "gelu"
IN_QUANT = None
OUT_QUANT = None
IN_DTYPES = {"x": "float32", "w_mat": "float32"}
WORLD_SIZE = 8

def setup_inputs():

    x = np.random.randn(8192, 8192).astype(np.float32)
    w_mat = (np.random.randn(8192, 8192) / np.sqrt(8192)).astype(np.float32)
    return {"x": x, "w_mat": w_mat}

def _epilogue(y):
    c = 0.7978845608028654
    return (0.5 * y * (1.0 + np.tanh(c * (y + 0.044715 * y ** 3)))).astype(np.float32)

def reference(x, w_mat):
    y = (x @ w_mat).astype(np.float32)
    return _epilogue(y)
```

```cpp
#include <hip/hip_runtime.h>
#include <rccl/rccl.h>
#include <cstdio>

#define MAINLOOP \
  "s_nop 4\n\t" \
  "v_mov_b32 v31, 0\n\t" \
  "v_mov_b32 v30, %[sCnt]\n\t" \
  "buffer_load_dword v27, v30, %[srdC], 0 offen sc1\n\t" \
  "buffer_load_dwordx4 v[204:207], %[vXa], %[srdX], %[sXL0] offen\n\t" \
  "buffer_load_dwordx4 v[208:211], %[vXa], %[srdX], %[sXL0] offen offset:64\n\t" \
  "buffer_load_dwordx4 v[212:215], %[vXa], %[srdX], %[sXL1] offen\n\t" \
  "buffer_load_dwordx4 v[216:219], %[vXa], %[srdX], %[sXL1] offen offset:64\n\t" \
  "v_add_u32 v16, 0x0, %[vW]\n\t" \
  "v_add_u32 v17, 0x8000, %[vW]\n\t" \
  "v_add_u32 v18, 0x10000, %[vW]\n\t" \
  "v_add_u32 v19, 0x18000, %[vW]\n\t" \
  "v_add_u32 v20, 0x80000, %[vW]\n\t" \
  "v_add_u32 v21, 0x88000, %[vW]\n\t" \
  "v_add_u32 v22, 0x90000, %[vW]\n\t" \
  "v_add_u32 v23, 0x98000, %[vW]\n\t" \
  "v_mov_b32 v25, %[vDS]\n\t" \
  "s_mov_b32 s43, %[sM0]\n\t" \
  "s_mov_b32 s48, %[sPH]\n\t" \
  "s_mov_b32 s49, 0\n\t" \
  "s_mov_b32 s69, 0\n\t" \
  "s_and_b32 s47, %[sK0], 0xffff\n\t" \
  "s_lshr_b32 s68, s47, 7\n\t" \
  "s_lshl_b32 s40, s68, 20\n\t" \
  "buffer_load_dwordx4 v[128:131], v16, %[srdW], s40 offen\n\t" \
  "buffer_load_dwordx4 v[132:135], v17, %[srdW], s40 offen\n\t" \
  "buffer_load_dwordx4 v[136:139], v18, %[srdW], s40 offen\n\t" \
  "buffer_load_dwordx4 v[140:143], v19, %[srdW], s40 offen\n\t" \
  "buffer_load_dwordx4 v[144:147], v20, %[srdW], s40 offen\n\t" \
  "buffer_load_dwordx4 v[148:151], v21, %[srdW], s40 offen\n\t" \
  "buffer_load_dwordx4 v[152:155], v22, %[srdW], s40 offen\n\t" \
  "buffer_load_dwordx4 v[156:159], v23, %[srdW], s40 offen\n\t" \
  "v_mov_b32 v28, 0\n\t" \
  "v_mov_b32 v29, 0\n\t" \
  "v_accvgpr_write_b32 a0, 0\n\t" \
  "v_accvgpr_write_b32 a1, 0\n\t" \
  "v_accvgpr_write_b32 a2, 0\n\t" \
  "v_accvgpr_write_b32 a3, 0\n\t" \
  "v_accvgpr_write_b32 a4, 0\n\t" \
  "v_accvgpr_write_b32 a5, 0\n\t" \
  "v_accvgpr_write_b32 a6, 0\n\t" \
  "v_accvgpr_write_b32 a7, 0\n\t" \
  "v_accvgpr_write_b32 a8, 0\n\t" \
  "v_accvgpr_write_b32 a9, 0\n\t" \
  "v_accvgpr_write_b32 a10, 0\n\t" \
  "v_accvgpr_write_b32 a11, 0\n\t" \
  "v_accvgpr_write_b32 a12, 0\n\t" \
  "v_accvgpr_write_b32 a13, 0\n\t" \
  "v_accvgpr_write_b32 a14, 0\n\t" \
  "v_accvgpr_write_b32 a15, 0\n\t" \
  "v_accvgpr_write_b32 a16, 0\n\t" \
  "v_accvgpr_write_b32 a17, 0\n\t" \
  "v_accvgpr_write_b32 a18, 0\n\t" \
  "v_accvgpr_write_b32 a19, 0\n\t" \
  "v_accvgpr_write_b32 a20, 0\n\t" \
  "v_accvgpr_write_b32 a21, 0\n\t" \
  "v_accvgpr_write_b32 a22, 0\n\t" \
  "v_accvgpr_write_b32 a23, 0\n\t" \
  "v_accvgpr_write_b32 a24, 0\n\t" \
  "v_accvgpr_write_b32 a25, 0\n\t" \
  "v_accvgpr_write_b32 a26, 0\n\t" \
  "v_accvgpr_write_b32 a27, 0\n\t" \
  "v_accvgpr_write_b32 a28, 0\n\t" \
  "v_accvgpr_write_b32 a29, 0\n\t" \
  "v_accvgpr_write_b32 a30, 0\n\t" \
  "v_accvgpr_write_b32 a31, 0\n\t" \
  "v_accvgpr_write_b32 a32, 0\n\t" \
  "v_accvgpr_write_b32 a33, 0\n\t" \
  "v_accvgpr_write_b32 a34, 0\n\t" \
  "v_accvgpr_write_b32 a35, 0\n\t" \
  "v_accvgpr_write_b32 a36, 0\n\t" \
  "v_accvgpr_write_b32 a37, 0\n\t" \
  "v_accvgpr_write_b32 a38, 0\n\t" \
  "v_accvgpr_write_b32 a39, 0\n\t" \
  "v_accvgpr_write_b32 a40, 0\n\t" \
  "v_accvgpr_write_b32 a41, 0\n\t" \
  "v_accvgpr_write_b32 a42, 0\n\t" \
  "v_accvgpr_write_b32 a43, 0\n\t" \
  "v_accvgpr_write_b32 a44, 0\n\t" \
  "v_accvgpr_write_b32 a45, 0\n\t" \
  "v_accvgpr_write_b32 a46, 0\n\t" \
  "v_accvgpr_write_b32 a47, 0\n\t" \
  "v_accvgpr_write_b32 a48, 0\n\t" \
  "v_accvgpr_write_b32 a49, 0\n\t" \
  "v_accvgpr_write_b32 a50, 0\n\t" \
  "v_accvgpr_write_b32 a51, 0\n\t" \
  "v_accvgpr_write_b32 a52, 0\n\t" \
  "v_accvgpr_write_b32 a53, 0\n\t" \
  "v_accvgpr_write_b32 a54, 0\n\t" \
  "v_accvgpr_write_b32 a55, 0\n\t" \
  "v_accvgpr_write_b32 a56, 0\n\t" \
  "v_accvgpr_write_b32 a57, 0\n\t" \
  "v_accvgpr_write_b32 a58, 0\n\t" \
  "v_accvgpr_write_b32 a59, 0\n\t" \
  "v_accvgpr_write_b32 a60, 0\n\t" \
  "v_accvgpr_write_b32 a61, 0\n\t" \
  "v_accvgpr_write_b32 a62, 0\n\t" \
  "v_accvgpr_write_b32 a63, 0\n\t" \
  "v_accvgpr_write_b32 a64, 0\n\t" \
  "v_accvgpr_write_b32 a65, 0\n\t" \
  "v_accvgpr_write_b32 a66, 0\n\t" \
  "v_accvgpr_write_b32 a67, 0\n\t" \
  "v_accvgpr_write_b32 a68, 0\n\t" \
  "v_accvgpr_write_b32 a69, 0\n\t" \
  "v_accvgpr_write_b32 a70, 0\n\t" \
  "v_accvgpr_write_b32 a71, 0\n\t" \
  "v_accvgpr_write_b32 a72, 0\n\t" \
  "v_accvgpr_write_b32 a73, 0\n\t" \
  "v_accvgpr_write_b32 a74, 0\n\t" \
  "v_accvgpr_write_b32 a75, 0\n\t" \
  "v_accvgpr_write_b32 a76, 0\n\t" \
  "v_accvgpr_write_b32 a77, 0\n\t" \
  "v_accvgpr_write_b32 a78, 0\n\t" \
  "v_accvgpr_write_b32 a79, 0\n\t" \
  "v_accvgpr_write_b32 a80, 0\n\t" \
  "v_accvgpr_write_b32 a81, 0\n\t" \
  "v_accvgpr_write_b32 a82, 0\n\t" \
  "v_accvgpr_write_b32 a83, 0\n\t" \
  "v_accvgpr_write_b32 a84, 0\n\t" \
  "v_accvgpr_write_b32 a85, 0\n\t" \
  "v_accvgpr_write_b32 a86, 0\n\t" \
  "v_accvgpr_write_b32 a87, 0\n\t" \
  "v_accvgpr_write_b32 a88, 0\n\t" \
  "v_accvgpr_write_b32 a89, 0\n\t" \
  "v_accvgpr_write_b32 a90, 0\n\t" \
  "v_accvgpr_write_b32 a91, 0\n\t" \
  "v_accvgpr_write_b32 a92, 0\n\t" \
  "v_accvgpr_write_b32 a93, 0\n\t" \
  "v_accvgpr_write_b32 a94, 0\n\t" \
  "v_accvgpr_write_b32 a95, 0\n\t" \
  "v_accvgpr_write_b32 a96, 0\n\t" \
  "v_accvgpr_write_b32 a97, 0\n\t" \
  "v_accvgpr_write_b32 a98, 0\n\t" \
  "v_accvgpr_write_b32 a99, 0\n\t" \
  "v_accvgpr_write_b32 a100, 0\n\t" \
  "v_accvgpr_write_b32 a101, 0\n\t" \
  "v_accvgpr_write_b32 a102, 0\n\t" \
  "v_accvgpr_write_b32 a103, 0\n\t" \
  "v_accvgpr_write_b32 a104, 0\n\t" \
  "v_accvgpr_write_b32 a105, 0\n\t" \
  "v_accvgpr_write_b32 a106, 0\n\t" \
  "v_accvgpr_write_b32 a107, 0\n\t" \
  "v_accvgpr_write_b32 a108, 0\n\t" \
  "v_accvgpr_write_b32 a109, 0\n\t" \
  "v_accvgpr_write_b32 a110, 0\n\t" \
  "v_accvgpr_write_b32 a111, 0\n\t" \
  "v_accvgpr_write_b32 a112, 0\n\t" \
  "v_accvgpr_write_b32 a113, 0\n\t" \
  "v_accvgpr_write_b32 a114, 0\n\t" \
  "v_accvgpr_write_b32 a115, 0\n\t" \
  "v_accvgpr_write_b32 a116, 0\n\t" \
  "v_accvgpr_write_b32 a117, 0\n\t" \
  "v_accvgpr_write_b32 a118, 0\n\t" \
  "v_accvgpr_write_b32 a119, 0\n\t" \
  "v_accvgpr_write_b32 a120, 0\n\t" \
  "v_accvgpr_write_b32 a121, 0\n\t" \
  "v_accvgpr_write_b32 a122, 0\n\t" \
  "v_accvgpr_write_b32 a123, 0\n\t" \
  "v_accvgpr_write_b32 a124, 0\n\t" \
  "v_accvgpr_write_b32 a125, 0\n\t" \
  "v_accvgpr_write_b32 a126, 0\n\t" \
  "v_accvgpr_write_b32 a127, 0\n\t" \
  "s_waitcnt vmcnt(8)\n\t" \
  "v_add_u32 v27, 1, v27\n\t" \
  "v_and_b32 v30, 1, v27\n\t" \
  "v_readfirstlane_b32 s59, v27\n\t" \
  "v_lshlrev_b32 v30, 24, v30\n\t" \
  "v_add_u32 v24, %[vA], v30\n\t" \
  "v_add_u32 v26, %[vSo], v30\n\t" \
  "v_cvt_pk_f16_f32 v220, v204, v205\n\t" \
  "v_cvt_pk_f16_f32 v221, v206, v207\n\t" \
  "v_cvt_pk_f16_f32 v222, v208, v209\n\t" \
  "v_cvt_pk_f16_f32 v223, v210, v211\n\t" \
  "v_cvt_pk_f16_f32 v224, v212, v213\n\t" \
  "v_cvt_pk_f16_f32 v225, v214, v215\n\t" \
  "v_cvt_pk_f16_f32 v226, v216, v217\n\t" \
  "v_cvt_pk_f16_f32 v227, v218, v219\n\t" \
  "buffer_store_dwordx4 v[220:223], v26, %[srdA], %[sAL0] offen sc0 sc1\n\t" \
  "buffer_store_dwordx4 v[224:227], v26, %[srdA], %[sAL1] offen sc0 sc1\n\t" \
  "v_accvgpr_write_b32 a128, 0\n\t" \
  "v_accvgpr_write_b32 a129, 0\n\t" \
  "v_accvgpr_write_b32 a130, 0\n\t" \
  "v_accvgpr_write_b32 a131, 0\n\t" \
  "v_accvgpr_write_b32 a132, 0\n\t" \
  "v_accvgpr_write_b32 a133, 0\n\t" \
  "v_accvgpr_write_b32 a134, 0\n\t" \
  "v_accvgpr_write_b32 a135, 0\n\t" \
  "v_accvgpr_write_b32 a136, 0\n\t" \
  "v_accvgpr_write_b32 a137, 0\n\t" \
  "v_accvgpr_write_b32 a138, 0\n\t" \
  "v_accvgpr_write_b32 a139, 0\n\t" \
  "v_accvgpr_write_b32 a140, 0\n\t" \
  "v_accvgpr_write_b32 a141, 0\n\t" \
  "v_accvgpr_write_b32 a142, 0\n\t" \
  "v_accvgpr_write_b32 a143, 0\n\t" \
  "v_accvgpr_write_b32 a144, 0\n\t" \
  "v_accvgpr_write_b32 a145, 0\n\t" \
  "v_accvgpr_write_b32 a146, 0\n\t" \
  "v_accvgpr_write_b32 a147, 0\n\t" \
  "v_accvgpr_write_b32 a148, 0\n\t" \
  "v_accvgpr_write_b32 a149, 0\n\t" \
  "v_accvgpr_write_b32 a150, 0\n\t" \
  "v_accvgpr_write_b32 a151, 0\n\t" \
  "v_accvgpr_write_b32 a152, 0\n\t" \
  "v_accvgpr_write_b32 a153, 0\n\t" \
  "v_accvgpr_write_b32 a154, 0\n\t" \
  "v_accvgpr_write_b32 a155, 0\n\t" \
  "v_accvgpr_write_b32 a156, 0\n\t" \
  "v_accvgpr_write_b32 a157, 0\n\t" \
  "v_accvgpr_write_b32 a158, 0\n\t" \
  "v_accvgpr_write_b32 a159, 0\n\t" \
  "v_accvgpr_write_b32 a160, 0\n\t" \
  "v_accvgpr_write_b32 a161, 0\n\t" \
  "v_accvgpr_write_b32 a162, 0\n\t" \
  "v_accvgpr_write_b32 a163, 0\n\t" \
  "v_accvgpr_write_b32 a164, 0\n\t" \
  "v_accvgpr_write_b32 a165, 0\n\t" \
  "v_accvgpr_write_b32 a166, 0\n\t" \
  "v_accvgpr_write_b32 a167, 0\n\t" \
  "v_accvgpr_write_b32 a168, 0\n\t" \
  "v_accvgpr_write_b32 a169, 0\n\t" \
  "v_accvgpr_write_b32 a170, 0\n\t" \
  "v_accvgpr_write_b32 a171, 0\n\t" \
  "v_accvgpr_write_b32 a172, 0\n\t" \
  "v_accvgpr_write_b32 a173, 0\n\t" \
  "v_accvgpr_write_b32 a174, 0\n\t" \
  "v_accvgpr_write_b32 a175, 0\n\t" \
  "v_accvgpr_write_b32 a176, 0\n\t" \
  "v_accvgpr_write_b32 a177, 0\n\t" \
  "v_accvgpr_write_b32 a178, 0\n\t" \
  "v_accvgpr_write_b32 a179, 0\n\t" \
  "v_accvgpr_write_b32 a180, 0\n\t" \
  "v_accvgpr_write_b32 a181, 0\n\t" \
  "v_accvgpr_write_b32 a182, 0\n\t" \
  "v_accvgpr_write_b32 a183, 0\n\t" \
  "v_accvgpr_write_b32 a184, 0\n\t" \
  "v_accvgpr_write_b32 a185, 0\n\t" \
  "v_accvgpr_write_b32 a186, 0\n\t" \
  "v_accvgpr_write_b32 a187, 0\n\t" \
  "v_accvgpr_write_b32 a188, 0\n\t" \
  "v_accvgpr_write_b32 a189, 0\n\t" \
  "v_accvgpr_write_b32 a190, 0\n\t" \
  "v_accvgpr_write_b32 a191, 0\n\t" \
  "v_accvgpr_write_b32 a192, 0\n\t" \
  "v_accvgpr_write_b32 a193, 0\n\t" \
  "v_accvgpr_write_b32 a194, 0\n\t" \
  "v_accvgpr_write_b32 a195, 0\n\t" \
  "v_accvgpr_write_b32 a196, 0\n\t" \
  "v_accvgpr_write_b32 a197, 0\n\t" \
  "v_accvgpr_write_b32 a198, 0\n\t" \
  "v_accvgpr_write_b32 a199, 0\n\t" \
  "v_accvgpr_write_b32 a200, 0\n\t" \
  "v_accvgpr_write_b32 a201, 0\n\t" \
  "v_accvgpr_write_b32 a202, 0\n\t" \
  "v_accvgpr_write_b32 a203, 0\n\t" \
  "v_accvgpr_write_b32 a204, 0\n\t" \
  "v_accvgpr_write_b32 a205, 0\n\t" \
  "v_accvgpr_write_b32 a206, 0\n\t" \
  "v_accvgpr_write_b32 a207, 0\n\t" \
  "v_accvgpr_write_b32 a208, 0\n\t" \
  "v_accvgpr_write_b32 a209, 0\n\t" \
  "v_accvgpr_write_b32 a210, 0\n\t" \
  "v_accvgpr_write_b32 a211, 0\n\t" \
  "v_accvgpr_write_b32 a212, 0\n\t" \
  "v_accvgpr_write_b32 a213, 0\n\t" \
  "v_accvgpr_write_b32 a214, 0\n\t" \
  "v_accvgpr_write_b32 a215, 0\n\t" \
  "v_accvgpr_write_b32 a216, 0\n\t" \
  "v_accvgpr_write_b32 a217, 0\n\t" \
  "v_accvgpr_write_b32 a218, 0\n\t" \
  "v_accvgpr_write_b32 a219, 0\n\t" \
  "v_accvgpr_write_b32 a220, 0\n\t" \
  "v_accvgpr_write_b32 a221, 0\n\t" \
  "v_accvgpr_write_b32 a222, 0\n\t" \
  "v_accvgpr_write_b32 a223, 0\n\t" \
  "v_accvgpr_write_b32 a224, 0\n\t" \
  "v_accvgpr_write_b32 a225, 0\n\t" \
  "v_accvgpr_write_b32 a226, 0\n\t" \
  "v_accvgpr_write_b32 a227, 0\n\t" \
  "v_accvgpr_write_b32 a228, 0\n\t" \
  "v_accvgpr_write_b32 a229, 0\n\t" \
  "v_accvgpr_write_b32 a230, 0\n\t" \
  "v_accvgpr_write_b32 a231, 0\n\t" \
  "v_accvgpr_write_b32 a232, 0\n\t" \
  "v_accvgpr_write_b32 a233, 0\n\t" \
  "v_accvgpr_write_b32 a234, 0\n\t" \
  "v_accvgpr_write_b32 a235, 0\n\t" \
  "v_accvgpr_write_b32 a236, 0\n\t" \
  "v_accvgpr_write_b32 a237, 0\n\t" \
  "v_accvgpr_write_b32 a238, 0\n\t" \
  "v_accvgpr_write_b32 a239, 0\n\t" \
  "v_accvgpr_write_b32 a240, 0\n\t" \
  "v_accvgpr_write_b32 a241, 0\n\t" \
  "v_accvgpr_write_b32 a242, 0\n\t" \
  "v_accvgpr_write_b32 a243, 0\n\t" \
  "v_accvgpr_write_b32 a244, 0\n\t" \
  "v_accvgpr_write_b32 a245, 0\n\t" \
  "v_accvgpr_write_b32 a246, 0\n\t" \
  "v_accvgpr_write_b32 a247, 0\n\t" \
  "v_accvgpr_write_b32 a248, 0\n\t" \
  "v_accvgpr_write_b32 a249, 0\n\t" \
  "v_accvgpr_write_b32 a250, 0\n\t" \
  "v_accvgpr_write_b32 a251, 0\n\t" \
  "v_accvgpr_write_b32 a252, 0\n\t" \
  "v_accvgpr_write_b32 a253, 0\n\t" \
  "v_accvgpr_write_b32 a254, 0\n\t" \
  "v_accvgpr_write_b32 a255, 0\n\t" \
  "s_waitcnt vmcnt(0)\n\t" \
  "s_barrier\n\t" \
  "s_mov_b64 exec, %[sLFm]\n\t" \
  "buffer_store_dword v27, %[vLF], %[srdC], 0 offen sc0 sc1\n\t" \
  "s_mov_b64 exec, -1\n\t" \
  "buffer_load_dwordx4 v[192:195], %[vXa], %[srdX], %[sXP] offen\n\t" \
  "buffer_load_dwordx4 v[196:199], %[vXa], %[srdX], %[sXP] offen offset:64\n\t" \
  "s_lshr_b32 s47, %[sK0], 16\n\t" \
  "s_lshr_b32 s68, s47, 7\n\t" \
  "s_lshl_b32 s40, s68, 20\n\t" \
  "buffer_load_dwordx4 v[160:163], v16, %[srdW], s40 offen\n\t" \
  "buffer_load_dwordx4 v[164:167], v17, %[srdW], s40 offen\n\t" \
  "buffer_load_dwordx4 v[168:171], v18, %[srdW], s40 offen\n\t" \
  "buffer_load_dwordx4 v[172:175], v19, %[srdW], s40 offen\n\t" \
  "buffer_load_dwordx4 v[176:179], v20, %[srdW], s40 offen\n\t" \
  "buffer_load_dwordx4 v[180:183], v21, %[srdW], s40 offen\n\t" \
  "buffer_load_dwordx4 v[184:187], v22, %[srdW], s40 offen\n\t" \
  "buffer_load_dwordx4 v[188:191], v23, %[srdW], s40 offen\n\t" \
  "s_and_b32 s47, %[sK0], 0xffff\n\t" \
  "s_lshr_b32 s68, s47, 7\n\t" \
  "s_lshl_b32 s42, s68, 16\n\t" \
  "s_and_b32 m0, s43, 0x1ffff\n\t" \
  "v_cmp_le_u32 s[50:51], s59, v28\n\t" \
  "v_cmp_le_u32 s[52:53], s59, v29\n\t" \
  "s_nop 1\n\t" \
  "s_bitcmp1_b32 s47, 6\n\ts_cselect_b64 s[78:79], s[52:53], s[50:51]\n\t" \
  "s_bitcmp1_b64 s[78:79], s47\n\ts_cbranch_scc0 LSLOWP0_%=\nLBACKP0_%=:\n\t" \
  "buffer_load_dwordx4 v24, %[srdA], s42 offen offset:0 sc0 sc1 lds\n\t" \
  "buffer_load_dwordx4 v24, %[srdA], s42 offen offset:1024 sc0 sc1 lds\n\t" \
  "buffer_load_dwordx4 v24, %[srdA], s42 offen offset:2048 sc0 sc1 lds\n\t" \
  "buffer_load_dwordx4 v24, %[srdA], s42 offen offset:3072 sc0 sc1 lds\n\t" \
  "s_lshr_b32 s47, %[sK0], 16\n\t" \
  "s_lshr_b32 s68, s47, 7\n\t" \
  "s_lshl_b32 s42, s68, 16\n\t" \
  "s_add_u32 s43, s43, 0x4000\n\t" \
  "s_and_b32 m0, s43, 0x1ffff\n\t" \
  "s_bitcmp1_b32 s47, 6\n\ts_cselect_b64 s[78:79], s[52:53], s[50:51]\n\t" \
  "s_bitcmp1_b64 s[78:79], s47\n\ts_cbranch_scc0 LSLOWP1_%=\nLBACKP1_%=:\n\t" \
  "buffer_load_dwordx4 v24, %[srdA], s42 offen offset:0 sc0 sc1 lds\n\t" \
  "buffer_load_dwordx4 v24, %[srdA], s42 offen offset:1024 sc0 sc1 lds\n\t" \
  "buffer_load_dwordx4 v24, %[srdA], s42 offen offset:2048 sc0 sc1 lds\n\t" \
  "buffer_load_dwordx4 v24, %[srdA], s42 offen offset:3072 sc0 sc1 lds\n\t" \
  "s_and_b32 s47, %[sK1], 0xffff\n\t" \
  "s_lshr_b32 s68, s47, 7\n\t" \
  "s_lshl_b32 s42, s68, 16\n\t" \
  "s_add_u32 s43, s43, 0x4000\n\t" \
  "s_and_b32 m0, s43, 0x1ffff\n\t" \
  "s_bitcmp1_b32 s47, 6\n\ts_cselect_b64 s[78:79], s[52:53], s[50:51]\n\t" \
  "s_bitcmp1_b64 s[78:79], s47\n\ts_cbranch_scc0 LSLOWP2_%=\nLBACKP2_%=:\n\t" \
  "buffer_load_dwordx4 v24, %[srdA], s42 offen offset:0 sc0 sc1 lds\n\t" \
  "buffer_load_dwordx4 v24, %[srdA], s42 offen offset:1024 sc0 sc1 lds\n\t" \
  "buffer_load_dwordx4 v24, %[srdA], s42 offen offset:2048 sc0 sc1 lds\n\t" \
  "buffer_load_dwordx4 v24, %[srdA], s42 offen offset:3072 sc0 sc1 lds\n\t" \
  "s_lshr_b32 s47, %[sK1], 16\n\t" \
  "s_lshr_b32 s68, s47, 7\n\t" \
  "s_lshl_b32 s42, s68, 16\n\t" \
  "s_add_u32 s43, s43, 0x4000\n\t" \
  "s_and_b32 m0, s43, 0x1ffff\n\t" \
  "s_bitcmp1_b32 s47, 6\n\ts_cselect_b64 s[78:79], s[52:53], s[50:51]\n\t" \
  "s_bitcmp1_b64 s[78:79], s47\n\ts_cbranch_scc0 LSLOWP3_%=\nLBACKP3_%=:\n\t" \
  "buffer_load_dwordx4 v24, %[srdA], s42 offen offset:0 sc0 sc1 lds\n\t" \
  "buffer_load_dwordx4 v24, %[srdA], s42 offen offset:1024 sc0 sc1 lds\n\t" \
  "buffer_load_dwordx4 v24, %[srdA], s42 offen offset:2048 sc0 sc1 lds\n\t" \
  "buffer_load_dwordx4 v24, %[srdA], s42 offen offset:3072 sc0 sc1 lds\n\t" \
  "s_waitcnt vmcnt(16)\n\t" \
  "s_and_b32 s47, %[sK1], 0xffff\n\t" \
  "s_lshr_b32 s68, s47, 7\n\t" \
  "s_lshl_b32 s40, s68, 20\n\t" \
  "v_cvt_pk_f16_f32 v64, v128, v132\n\t" \
  "v_cvt_pk_f16_f32 v68, v129, v133\n\t" \
  "v_cvt_pk_f16_f32 v72, v130, v134\n\t" \
  "v_cvt_pk_f16_f32 v76, v131, v135\n\t" \
  "buffer_load_dwordx4 v[128:131], v16, %[srdW], s40 offen\n\t" \
  "buffer_load_dwordx4 v[132:135], v17, %[srdW], s40 offen\n\t" \
  "v_cvt_pk_f16_f32 v65, v136, v140\n\t" \
  "v_cvt_pk_f16_f32 v69, v137, v141\n\t" \
  "v_cvt_pk_f16_f32 v73, v138, v142\n\t" \
  "v_cvt_pk_f16_f32 v77, v139, v143\n\t" \
  "buffer_load_dwordx4 v[136:139], v18, %[srdW], s40 offen\n\t" \
  "buffer_load_dwordx4 v[140:143], v19, %[srdW], s40 offen\n\t" \
  "v_cvt_pk_f16_f32 v66, v144, v148\n\t" \
  "v_cvt_pk_f16_f32 v70, v145, v149\n\t" \
  "v_cvt_pk_f16_f32 v74, v146, v150\n\t" \
  "v_cvt_pk_f16_f32 v78, v147, v151\n\t" \
  "buffer_load_dwordx4 v[144:147], v20, %[srdW], s40 offen\n\t" \
  "buffer_load_dwordx4 v[148:151], v21, %[srdW], s40 offen\n\t" \
  "v_cvt_pk_f16_f32 v67, v152, v156\n\t" \
  "v_cvt_pk_f16_f32 v71, v153, v157\n\t" \
  "v_cvt_pk_f16_f32 v75, v154, v158\n\t" \
  "v_cvt_pk_f16_f32 v79, v155, v159\n\t" \
  "buffer_load_dwordx4 v[152:155], v22, %[srdW], s40 offen\n\t" \
  "buffer_load_dwordx4 v[156:159], v23, %[srdW], s40 offen\n\t" \
  "s_lshr_b32 s47, %[sK1], 16\n\t" \
  "s_lshr_b32 s68, s47, 7\n\t" \
  "s_lshl_b32 s40, s68, 20\n\t" \
  "v_cvt_pk_f16_f32 v80, v160, v164\n\t" \
  "v_cvt_pk_f16_f32 v84, v161, v165\n\t" \
  "v_cvt_pk_f16_f32 v88, v162, v166\n\t" \
  "v_cvt_pk_f16_f32 v92, v163, v167\n\t" \
  "buffer_load_dwordx4 v[160:163], v16, %[srdW], s40 offen\n\t" \
  "buffer_load_dwordx4 v[164:167], v17, %[srdW], s40 offen\n\t" \
  "v_cvt_pk_f16_f32 v81, v168, v172\n\t" \
  "v_cvt_pk_f16_f32 v85, v169, v173\n\t" \
  "v_cvt_pk_f16_f32 v89, v170, v174\n\t" \
  "v_cvt_pk_f16_f32 v93, v171, v175\n\t" \
  "buffer_load_dwordx4 v[168:171], v18, %[srdW], s40 offen\n\t" \
  "buffer_load_dwordx4 v[172:175], v19, %[srdW], s40 offen\n\t" \
  "v_cvt_pk_f16_f32 v82, v176, v180\n\t" \
  "v_cvt_pk_f16_f32 v86, v177, v181\n\t" \
  "v_cvt_pk_f16_f32 v90, v178, v182\n\t" \
  "v_cvt_pk_f16_f32 v94, v179, v183\n\t" \
  "buffer_load_dwordx4 v[176:179], v20, %[srdW], s40 offen\n\t" \
  "buffer_load_dwordx4 v[180:183], v21, %[srdW], s40 offen\n\t" \
  "v_cvt_pk_f16_f32 v83, v184, v188\n\t" \
  "v_cvt_pk_f16_f32 v87, v185, v189\n\t" \
  "v_cvt_pk_f16_f32 v91, v186, v190\n\t" \
  "v_cvt_pk_f16_f32 v95, v187, v191\n\t" \
  "buffer_load_dwordx4 v[184:187], v22, %[srdW], s40 offen\n\t" \
  "buffer_load_dwordx4 v[188:191], v23, %[srdW], s40 offen\n\t" \
  "s_mov_b32 s44, 0\n\t" \
  "v_readlane_b32 s46, %[vTab], s44\n\t" \
  "s_nop 1\n\t" \
  "s_add_u32 s44, s44, 1\n\t" \
  "s_nop 1\n\t" \
  "v_cmp_le_u32 s[50:51], s59, v28\n\t" \
  "s_nop 1\n\t" \
  "v_cmp_le_u32 s[52:53], s59, v29\n\t" \
  "s_nop 1\n\t" \
  "s_and_b32 s47, s46, 0xffff\n\t" \
  "s_nop 1\n\t" \
  "s_bitcmp1_b32 s47, 6\n\ts_cselect_b64 s[78:79], s[52:53], s[50:51]\n\t" \
  "s_nop 1\n\t" \
  "s_lshr_b32 s40, s47, 7\n\t" \
  "s_nop 1\n\t" \
  "s_lshl_b32 s42, s40, 16\n\t" \
  "s_nop 1\n\t" \
  "s_lshl_b32 s40, s40, 20\n\t" \
  "s_nop 1\n\t" \
  "s_add_u32 s43, s43, 0x4000\n\t" \
  "s_nop 1\n\t" \
  "s_and_b32 m0, s43, 0x1ffff\n\t" \
  "s_nop 1\n\t" \
  "s_bitcmp1_b64 s[78:79], s47\n\ts_cbranch_scc0 LSLOWP4_%=\nLBACKP4_%=:\n\t" \
  "s_nop 1\n\t" \
  "s_waitcnt vmcnt(16)\n\t" \
  "s_barrier\n\t" \
  "ds_read_b128 v[32:35], v25 offset:0\n\t" \
  "ds_read_b128 v[36:39], v25 offset:16384\n\t" \
  "ds_read_b128 v[40:43], v25 offset:1024\n\t" \
  "ds_read_b128 v[44:47], v25 offset:17408\n\t" \
  "s_waitcnt vmcnt(0)\n\t" \
  "s_mov_b32 s45, 32\n\t" \
  ".p2align 6\n\t" \
  "LN0_%=:\n\t" \
  "s_cmp_eq_u32 s48, 0\n\t" \
  "s_cbranch_scc1 LC0_%=\n\t" \
  "s_waitcnt lgkmcnt(2)\n\t" \
  "v_mfma_f32_16x16x32_f16 a[0:3], v[32:35], v[64:67], a[0:3]\n\t" \
  "ds_read_b128 v[48:51], v25 offset:2048\n\t" \
  "v_mfma_f32_16x16x32_f16 a[0:3], v[36:39], v[80:83], a[0:3]\n\t" \
  "v_mfma_f32_16x16x32_f16 a[4:7], v[36:39], v[84:87], a[4:7]\n\t" \
  "buffer_load_dwordx4 v24, %[srdA], s42 offen offset:0 sc0 sc1 lds\n\t" \
  "v_mfma_f32_16x16x32_f16 a[4:7], v[32:35], v[68:71], a[4:7]\n\t" \
  "v_mfma_f32_16x16x32_f16 a[8:11], v[32:35], v[72:75], a[8:11]\n\t" \
  "ds_read_b128 v[52:55], v25 offset:18432\n\t" \
  "v_mfma_f32_16x16x32_f16 a[8:11], v[36:39], v[88:91], a[8:11]\n\t" \
  "v_mfma_f32_16x16x32_f16 a[12:15], v[36:39], v[92:95], a[12:15]\n\t" \
  "buffer_load_dwordx4 v24, %[srdA], s42 offen offset:1024 sc0 sc1 lds\n\t" \
  "v_mfma_f32_16x16x32_f16 a[12:15], v[32:35], v[76:79], a[12:15]\n\t" \
  "s_waitcnt lgkmcnt(2)\n\t" \
  "v_mfma_f32_16x16x32_f16 a[28:31], v[40:43], v[76:79], a[28:31]\n\t" \
  "ds_read_b128 v[56:59], v25 offset:3072\n\t" \
  "v_mfma_f32_16x16x32_f16 a[28:31], v[44:47], v[92:95], a[28:31]\n\t" \
  "v_mfma_f32_16x16x32_f16 a[24:27], v[44:47], v[88:91], a[24:27]\n\t" \
  "buffer_load_dwordx4 v24, %[srdA], s42 offen offset:2048 sc0 sc1 lds\n\t" \
  "v_mfma_f32_16x16x32_f16 a[24:27], v[40:43], v[72:75], a[24:27]\n\t" \
  "s_waitcnt vmcnt(21)\n\t" \
  "v_cvt_pk_f16_f32 v96, v128, v132\n\t" \
  "v_mfma_f32_16x16x32_f16 a[20:23], v[40:43], v[68:71], a[20:23]\n\t" \
  "ds_read_b128 v[60:63], v25 offset:19456\n\t" \
  "v_mfma_f32_16x16x32_f16 a[20:23], v[44:47], v[84:87], a[20:23]\n\t" \
  "v_cvt_pk_f16_f32 v100, v129, v133\n\t" \
  "v_mfma_f32_16x16x32_f16 a[16:19], v[44:47], v[80:83], a[16:19]\n\t" \
  "buffer_load_dwordx4 v24, %[srdA], s42 offen offset:3072 sc0 sc1 lds\n\t" \
  "v_mfma_f32_16x16x32_f16 a[16:19], v[40:43], v[64:67], a[16:19]\n\t" \
  "v_cvt_pk_f16_f32 v104, v130, v134\n\t" \
  "s_waitcnt lgkmcnt(2)\n\t" \
  "v_mfma_f32_16x16x32_f16 a[32:35], v[48:51], v[64:67], a[32:35]\n\t" \
  "ds_read_b128 v[32:35], v25 offset:4096\n\t" \
  "v_mfma_f32_16x16x32_f16 a[32:35], v[52:55], v[80:83], a[32:35]\n\t" \
  "v_cvt_pk_f16_f32 v108, v131, v135\n\t" \
  "v_mfma_f32_16x16x32_f16 a[36:39], v[52:55], v[84:87], a[36:39]\n\t" \
  "buffer_load_dwordx4 v[128:131], v16, %[srdW], s40 offen\n\t" \
  "v_mfma_f32_16x16x32_f16 a[36:39], v[48:51], v[68:71], a[36:39]\n\t" \
  "s_waitcnt vmcnt(21)\n\t" \
  "v_cvt_pk_f16_f32 v97, v136, v140\n\t" \
  "v_mfma_f32_16x16x32_f16 a[40:43], v[48:51], v[72:75], a[40:43]\n\t" \
  "ds_read_b128 v[36:39], v25 offset:20480\n\t" \
  "v_mfma_f32_16x16x32_f16 a[40:43], v[52:55], v[88:91], a[40:43]\n\t" \
  "v_cvt_pk_f16_f32 v101, v137, v141\n\t" \
  "v_mfma_f32_16x16x32_f16 a[44:47], v[52:55], v[92:95], a[44:47]\n\t" \
  "buffer_load_dwordx4 v[132:135], v17, %[srdW], s40 offen\n\t" \
  "v_mfma_f32_16x16x32_f16 a[44:47], v[48:51], v[76:79], a[44:47]\n\t" \
  "v_cvt_pk_f16_f32 v105, v138, v142\n\t" \
  "s_waitcnt lgkmcnt(2)\n\t" \
  "v_mfma_f32_16x16x32_f16 a[60:63], v[56:59], v[76:79], a[60:63]\n\t" \
  "ds_read_b128 v[40:43], v25 offset:5120\n\t" \
  "v_mfma_f32_16x16x32_f16 a[60:63], v[60:63], v[92:95], a[60:63]\n\t" \
  "v_cvt_pk_f16_f32 v109, v139, v143\n\t" \
  "v_mfma_f32_16x16x32_f16 a[56:59], v[60:63], v[88:91], a[56:59]\n\t" \
  "buffer_load_dwordx4 v[136:139], v18, %[srdW], s40 offen\n\t" \
  "v_mfma_f32_16x16x32_f16 a[56:59], v[56:59], v[72:75], a[56:59]\n\t" \
  "s_waitcnt vmcnt(21)\n\t" \
  "v_cvt_pk_f16_f32 v98, v144, v148\n\t" \
  "v_mfma_f32_16x16x32_f16 a[52:55], v[56:59], v[68:71], a[52:55]\n\t" \
  "ds_read_b128 v[44:47], v25 offset:21504\n\t" \
  "v_mfma_f32_16x16x32_f16 a[52:55], v[60:63], v[84:87], a[52:55]\n\t" \
  "v_cvt_pk_f16_f32 v102, v145, v149\n\t" \
  "v_mfma_f32_16x16x32_f16 a[48:51], v[60:63], v[80:83], a[48:51]\n\t" \
  "buffer_load_dwordx4 v[140:143], v19, %[srdW], s40 offen\n\t" \
  "v_mfma_f32_16x16x32_f16 a[48:51], v[56:59], v[64:67], a[48:51]\n\t" \
  "v_cvt_pk_f16_f32 v106, v146, v150\n\t" \
  "s_waitcnt lgkmcnt(2)\n\t" \
  "v_mfma_f32_16x16x32_f16 a[64:67], v[32:35], v[64:67], a[64:67]\n\t" \
  "ds_read_b128 v[48:51], v25 offset:6144\n\t" \
  "v_mfma_f32_16x16x32_f16 a[64:67], v[36:39], v[80:83], a[64:67]\n\t" \
  "v_cvt_pk_f16_f32 v110, v147, v151\n\t" \
  "v_mfma_f32_16x16x32_f16 a[68:71], v[36:39], v[84:87], a[68:71]\n\t" \
  "buffer_load_dwordx4 v[144:147], v20, %[srdW], s40 offen\n\t" \
  "v_mfma_f32_16x16x32_f16 a[68:71], v[32:35], v[68:71], a[68:71]\n\t" \
  "s_waitcnt vmcnt(21)\n\t" \
  "v_cvt_pk_f16_f32 v99, v152, v156\n\t" \
  "v_mfma_f32_16x16x32_f16 a[72:75], v[32:35], v[72:75], a[72:75]\n\t" \
  "ds_read_b128 v[52:55], v25 offset:22528\n\t" \
  "v_mfma_f32_16x16x32_f16 a[72:75], v[36:39], v[88:91], a[72:75]\n\t" \
  "v_cvt_pk_f16_f32 v103, v153, v157\n\t" \
  "v_mfma_f32_16x16x32_f16 a[76:79], v[36:39], v[92:95], a[76:79]\n\t" \
  "buffer_load_dwordx4 v[148:151], v21, %[srdW], s40 offen\n\t" \
  "v_mfma_f32_16x16x32_f16 a[76:79], v[32:35], v[76:79], a[76:79]\n\t" \
  "v_cvt_pk_f16_f32 v107, v154, v158\n\t" \
  "s_waitcnt lgkmcnt(2)\n\t" \
  "v_mfma_f32_16x16x32_f16 a[92:95], v[40:43], v[76:79], a[92:95]\n\t" \
  "ds_read_b128 v[56:59], v25 offset:7168\n\t" \
  "v_mfma_f32_16x16x32_f16 a[92:95], v[44:47], v[92:95], a[92:95]\n\t" \
  "v_cvt_pk_f16_f32 v111, v155, v159\n\t" \
  "v_mfma_f32_16x16x32_f16 a[88:91], v[44:47], v[88:91], a[88:91]\n\t" \
  "buffer_load_dwordx4 v[152:155], v22, %[srdW], s40 offen\n\t" \
  "v_mfma_f32_16x16x32_f16 a[88:91], v[40:43], v[72:75], a[88:91]\n\t" \
  "v_mfma_f32_16x16x32_f16 a[84:87], v[40:43], v[68:71], a[84:87]\n\t" \
  "ds_read_b128 v[60:63], v25 offset:23552\n\t" \
  "v_mfma_f32_16x16x32_f16 a[84:87], v[44:47], v[84:87], a[84:87]\n\t" \
  "v_mfma_f32_16x16x32_f16 a[80:83], v[44:47], v[80:83], a[80:83]\n\t" \
  "buffer_load_dwordx4 v[156:159], v23, %[srdW], s40 offen\n\t" \
  "v_mfma_f32_16x16x32_f16 a[80:83], v[40:43], v[64:67], a[80:83]\n\t" \
  "s_waitcnt lgkmcnt(2)\n\t" \
  "v_mfma_f32_16x16x32_f16 a[96:99], v[48:51], v[64:67], a[96:99]\n\t" \
  "ds_read_b128 v[32:35], v25 offset:8192\n\t" \
  "v_mfma_f32_16x16x32_f16 a[96:99], v[52:55], v[80:83], a[96:99]\n\t" \
  "v_cmp_le_u32 s[50:51], s59, v28\n\t" \
  "v_mfma_f32_16x16x32_f16 a[100:103], v[52:55], v[84:87], a[100:103]\n\t" \
  "v_cmp_le_u32 s[52:53], s59, v29\n\t" \
  "v_mfma_f32_16x16x32_f16 a[100:103], v[48:51], v[68:71], a[100:103]\n\t" \
  "s_lshr_b32 s47, s46, 16\n\t" \
  "v_mfma_f32_16x16x32_f16 a[104:107], v[48:51], v[72:75], a[104:107]\n\t" \
  "ds_read_b128 v[36:39], v25 offset:24576\n\t" \
  "v_mfma_f32_16x16x32_f16 a[104:107], v[52:55], v[88:91], a[104:107]\n\t" \
  "s_bitcmp1_b32 s47, 6\n\ts_cselect_b64 s[78:79], s[52:53], s[50:51]\n\t" \
  "v_mfma_f32_16x16x32_f16 a[108:111], v[52:55], v[92:95], a[108:111]\n\t" \
  "s_lshr_b32 s40, s47, 7\n\t" \
  "v_mfma_f32_16x16x32_f16 a[108:111], v[48:51], v[76:79], a[108:111]\n\t" \
  "s_lshl_b32 s42, s40, 16\n\t" \
  "s_waitcnt lgkmcnt(2)\n\t" \
  "v_mfma_f32_16x16x32_f16 a[124:127], v[56:59], v[76:79], a[124:127]\n\t" \
  "ds_read_b128 v[40:43], v25 offset:9216\n\t" \
  "v_mfma_f32_16x16x32_f16 a[124:127], v[60:63], v[92:95], a[124:127]\n\t" \
  "s_lshl_b32 s40, s40, 20\n\t" \
  "v_mfma_f32_16x16x32_f16 a[120:123], v[60:63], v[88:91], a[120:123]\n\t" \
  "s_add_u32 s43, s43, 0x4000\n\t" \
  "v_mfma_f32_16x16x32_f16 a[120:123], v[56:59], v[72:75], a[120:123]\n\t" \
  "s_and_b32 m0, s43, 0x1ffff\n\t" \
  "v_mfma_f32_16x16x32_f16 a[116:119], v[56:59], v[68:71], a[116:119]\n\t" \
  "ds_read_b128 v[44:47], v25 offset:25600\n\t" \
  "v_mfma_f32_16x16x32_f16 a[116:119], v[60:63], v[84:87], a[116:119]\n\t" \
  "s_bitcmp1_b64 s[78:79], s47\n\ts_cbranch_scc0 LSLOWN0_%=\nLBACKN0_%=:\n\t" \
  "v_mfma_f32_16x16x32_f16 a[112:115], v[60:63], v[80:83], a[112:115]\n\t" \
  "v_mfma_f32_16x16x32_f16 a[112:115], v[56:59], v[64:67], a[112:115]\n\t" \
  "LN1_%=:\n\t" \
  "s_cmp_eq_u32 s48, 1\n\t" \
  "s_cbranch_scc1 LC1_%=\n\t" \
  "s_waitcnt lgkmcnt(2)\n\t" \
  "v_mfma_f32_16x16x32_f16 a[128:131], v[32:35], v[64:67], a[128:131]\n\t" \
  "ds_read_b128 v[48:51], v25 offset:10240\n\t" \
  "v_mfma_f32_16x16x32_f16 a[128:131], v[36:39], v[80:83], a[128:131]\n\t" \
  "v_mfma_f32_16x16x32_f16 a[132:135], v[36:39], v[84:87], a[132:135]\n\t" \
  "buffer_load_dwordx4 v24, %[srdA], s42 offen offset:0 sc0 sc1 lds\n\t" \
  "v_mfma_f32_16x16x32_f16 a[132:135], v[32:35], v[68:71], a[132:135]\n\t" \
  "v_mfma_f32_16x16x32_f16 a[136:139], v[32:35], v[72:75], a[136:139]\n\t" \
  "ds_read_b128 v[52:55], v25 offset:26624\n\t" \
  "v_mfma_f32_16x16x32_f16 a[136:139], v[36:39], v[88:91], a[136:139]\n\t" \
  "v_mfma_f32_16x16x32_f16 a[140:143], v[36:39], v[92:95], a[140:143]\n\t" \
  "buffer_load_dwordx4 v24, %[srdA], s42 offen offset:1024 sc0 sc1 lds\n\t" \
  "v_mfma_f32_16x16x32_f16 a[140:143], v[32:35], v[76:79], a[140:143]\n\t" \
  "s_waitcnt lgkmcnt(2)\n\t" \
  "v_mfma_f32_16x16x32_f16 a[156:159], v[40:43], v[76:79], a[156:159]\n\t" \
  "ds_read_b128 v[56:59], v25 offset:11264\n\t" \
  "v_mfma_f32_16x16x32_f16 a[156:159], v[44:47], v[92:95], a[156:159]\n\t" \
  "v_mfma_f32_16x16x32_f16 a[152:155], v[44:47], v[88:91], a[152:155]\n\t" \
  "buffer_load_dwordx4 v24, %[srdA], s42 offen offset:2048 sc0 sc1 lds\n\t" \
  "v_mfma_f32_16x16x32_f16 a[152:155], v[40:43], v[72:75], a[152:155]\n\t" \
  "s_waitcnt vmcnt(21)\n\t" \
  "s_barrier\n\t" \
  "v_cvt_pk_f16_f32 v112, v160, v164\n\t" \
  "v_mfma_f32_16x16x32_f16 a[148:151], v[40:43], v[68:71], a[148:151]\n\t" \
  "ds_read_b128 v[60:63], v25 offset:27648\n\t" \
  "v_mfma_f32_16x16x32_f16 a[148:151], v[44:47], v[84:87], a[148:151]\n\t" \
  "v_cvt_pk_f16_f32 v116, v161, v165\n\t" \
  "v_mfma_f32_16x16x32_f16 a[144:147], v[44:47], v[80:83], a[144:147]\n\t" \
  "buffer_load_dwordx4 v24, %[srdA], s42 offen offset:3072 sc0 sc1 lds\n\t" \
  "v_mfma_f32_16x16x32_f16 a[144:147], v[40:43], v[64:67], a[144:147]\n\t" \
  "v_cvt_pk_f16_f32 v120, v162, v166\n\t" \
  "s_waitcnt lgkmcnt(2)\n\t" \
  "v_mfma_f32_16x16x32_f16 a[160:163], v[48:51], v[64:67], a[160:163]\n\t" \
  "ds_read_b128 v[32:35], v25 offset:12288\n\t" \
  "v_mfma_f32_16x16x32_f16 a[160:163], v[52:55], v[80:83], a[160:163]\n\t" \
  "v_cvt_pk_f16_f32 v124, v163, v167\n\t" \
  "v_mfma_f32_16x16x32_f16 a[164:167], v[52:55], v[84:87], a[164:167]\n\t" \
  "buffer_load_dwordx4 v[160:163], v16, %[srdW], s40 offen\n\t" \
  "v_mfma_f32_16x16x32_f16 a[164:167], v[48:51], v[68:71], a[164:167]\n\t" \
  "s_waitcnt vmcnt(21)\n\t" \
  "v_cvt_pk_f16_f32 v113, v168, v172\n\t" \
  "v_mfma_f32_16x16x32_f16 a[168:171], v[48:51], v[72:75], a[168:171]\n\t" \
  "ds_read_b128 v[36:39], v25 offset:28672\n\t" \
  "v_mfma_f32_16x16x32_f16 a[168:171], v[52:55], v[88:91], a[168:171]\n\t" \
  "v_cvt_pk_f16_f32 v117, v169, v173\n\t" \
  "v_mfma_f32_16x16x32_f16 a[172:175], v[52:55], v[92:95], a[172:175]\n\t" \
  "buffer_load_dwordx4 v[164:167], v17, %[srdW], s40 offen\n\t" \
  "v_mfma_f32_16x16x32_f16 a[172:175], v[48:51], v[76:79], a[172:175]\n\t" \
  "v_cvt_pk_f16_f32 v121, v170, v174\n\t" \
  "s_waitcnt lgkmcnt(2)\n\t" \
  "v_mfma_f32_16x16x32_f16 a[188:191], v[56:59], v[76:79], a[188:191]\n\t" \
  "ds_read_b128 v[40:43], v25 offset:13312\n\t" \
  "v_mfma_f32_16x16x32_f16 a[188:191], v[60:63], v[92:95], a[188:191]\n\t" \
  "v_cvt_pk_f16_f32 v125, v171, v175\n\t" \
  "v_mfma_f32_16x16x32_f16 a[184:187], v[60:63], v[88:91], a[184:187]\n\t" \
  "buffer_load_dwordx4 v[168:171], v18, %[srdW], s40 offen\n\t" \
  "v_mfma_f32_16x16x32_f16 a[184:187], v[56:59], v[72:75], a[184:187]\n\t" \
  "s_waitcnt vmcnt(21)\n\t" \
  "v_cvt_pk_f16_f32 v114, v176, v180\n\t" \
  "v_mfma_f32_16x16x32_f16 a[180:183], v[56:59], v[68:71], a[180:183]\n\t" \
  "ds_read_b128 v[44:47], v25 offset:29696\n\t" \
  "v_mfma_f32_16x16x32_f16 a[180:183], v[60:63], v[84:87], a[180:183]\n\t" \
  "v_cvt_pk_f16_f32 v118, v177, v181\n\t" \
  "v_mfma_f32_16x16x32_f16 a[176:179], v[60:63], v[80:83], a[176:179]\n\t" \
  "buffer_load_dwordx4 v[172:175], v19, %[srdW], s40 offen\n\t" \
  "v_mfma_f32_16x16x32_f16 a[176:179], v[56:59], v[64:67], a[176:179]\n\t" \
  "v_cvt_pk_f16_f32 v122, v178, v182\n\t" \
  "s_waitcnt lgkmcnt(2)\n\t" \
  "v_mfma_f32_16x16x32_f16 a[192:195], v[32:35], v[64:67], a[192:195]\n\t" \
  "ds_read_b128 v[48:51], v25 offset:14336\n\t" \
  "v_mfma_f32_16x16x32_f16 a[192:195], v[36:39], v[80:83], a[192:195]\n\t" \
  "v_cvt_pk_f16_f32 v126, v179, v183\n\t" \
  "v_mfma_f32_16x16x32_f16 a[196:199], v[36:39], v[84:87], a[196:199]\n\t" \
  "buffer_load_dwordx4 v[176:179], v20, %[srdW], s40 offen\n\t" \
  "v_mfma_f32_16x16x32_f16 a[196:199], v[32:35], v[68:71], a[196:199]\n\t" \
  "s_waitcnt vmcnt(21)\n\t" \
  "v_cvt_pk_f16_f32 v115, v184, v188\n\t" \
  "v_mfma_f32_16x16x32_f16 a[200:203], v[32:35], v[72:75], a[200:203]\n\t" \
  "ds_read_b128 v[52:55], v25 offset:30720\n\t" \
  "v_mfma_f32_16x16x32_f16 a[200:203], v[36:39], v[88:91], a[200:203]\n\t" \
  "v_cvt_pk_f16_f32 v119, v185, v189\n\t" \
  "v_mfma_f32_16x16x32_f16 a[204:207], v[36:39], v[92:95], a[204:207]\n\t" \
  "buffer_load_dwordx4 v[180:183], v21, %[srdW], s40 offen\n\t" \
  "v_mfma_f32_16x16x32_f16 a[204:207], v[32:35], v[76:79], a[204:207]\n\t" \
  "v_cvt_pk_f16_f32 v123, v186, v190\n\t" \
  "s_waitcnt lgkmcnt(2)\n\t" \
  "v_mfma_f32_16x16x32_f16 a[220:223], v[40:43], v[76:79], a[220:223]\n\t" \
  "ds_read_b128 v[56:59], v25 offset:15360\n\t" \
  "v_mfma_f32_16x16x32_f16 a[220:223], v[44:47], v[92:95], a[220:223]\n\t" \
  "v_cvt_pk_f16_f32 v127, v187, v191\n\t" \
  "v_mfma_f32_16x16x32_f16 a[216:219], v[44:47], v[88:91], a[216:219]\n\t" \
  "buffer_load_dwordx4 v[184:187], v22, %[srdW], s40 offen\n\t" \
  "v_mfma_f32_16x16x32_f16 a[216:219], v[40:43], v[72:75], a[216:219]\n\t" \
  "v_mfma_f32_16x16x32_f16 a[212:215], v[40:43], v[68:71], a[212:215]\n\t" \
  "ds_read_b128 v[60:63], v25 offset:31744\n\t" \
  "v_mfma_f32_16x16x32_f16 a[212:215], v[44:47], v[84:87], a[212:215]\n\t" \
  "v_readlane_b32 s46, %[vTab], s44\n\t" \
  "v_mfma_f32_16x16x32_f16 a[208:211], v[44:47], v[80:83], a[208:211]\n\t" \
  "buffer_load_dwordx4 v[188:191], v23, %[srdW], s40 offen\n\t" \
  "v_mfma_f32_16x16x32_f16 a[208:211], v[40:43], v[64:67], a[208:211]\n\t" \
  "s_add_u32 s44, s44, 1\n\t" \
  "s_waitcnt lgkmcnt(2)\n\t" \
  "v_mfma_f32_16x16x32_f16 a[224:227], v[48:51], v[64:67], a[224:227]\n\t" \
  "ds_read_b128 v[32:35], v25 offset:32768\n\t" \
  "v_mfma_f32_16x16x32_f16 a[224:227], v[52:55], v[80:83], a[224:227]\n\t" \
  "v_cmp_le_u32 s[50:51], s59, v28\n\t" \
  "v_mfma_f32_16x16x32_f16 a[228:231], v[52:55], v[84:87], a[228:231]\n\t" \
  "v_cmp_le_u32 s[52:53], s59, v29\n\t" \
  "v_mfma_f32_16x16x32_f16 a[228:231], v[48:51], v[68:71], a[228:231]\n\t" \
  "s_and_b32 s47, s46, 0xffff\n\t" \
  "v_mfma_f32_16x16x32_f16 a[232:235], v[48:51], v[72:75], a[232:235]\n\t" \
  "ds_read_b128 v[36:39], v25 offset:49152\n\t" \
  "v_mfma_f32_16x16x32_f16 a[232:235], v[52:55], v[88:91], a[232:235]\n\t" \
  "s_bitcmp1_b32 s47, 6\n\ts_cselect_b64 s[78:79], s[52:53], s[50:51]\n\t" \
  "v_mfma_f32_16x16x32_f16 a[236:239], v[52:55], v[92:95], a[236:239]\n\t" \
  "s_lshr_b32 s40, s47, 7\n\t" \
  "v_mfma_f32_16x16x32_f16 a[236:239], v[48:51], v[76:79], a[236:239]\n\t" \
  "s_lshl_b32 s42, s40, 16\n\t" \
  "s_waitcnt lgkmcnt(2)\n\t" \
  "v_mfma_f32_16x16x32_f16 a[252:255], v[56:59], v[76:79], a[252:255]\n\t" \
  "ds_read_b128 v[40:43], v25 offset:33792\n\t" \
  "v_mfma_f32_16x16x32_f16 a[252:255], v[60:63], v[92:95], a[252:255]\n\t" \
  "s_lshl_b32 s40, s40, 20\n\t" \
  "v_mfma_f32_16x16x32_f16 a[248:251], v[60:63], v[88:91], a[248:251]\n\t" \
  "s_add_u32 s43, s43, 0x4000\n\t" \
  "v_mfma_f32_16x16x32_f16 a[248:251], v[56:59], v[72:75], a[248:251]\n\t" \
  "s_and_b32 m0, s43, 0x1ffff\n\t" \
  "v_mfma_f32_16x16x32_f16 a[244:247], v[56:59], v[68:71], a[244:247]\n\t" \
  "ds_read_b128 v[44:47], v25 offset:50176\n\t" \
  "v_mfma_f32_16x16x32_f16 a[244:247], v[60:63], v[84:87], a[244:247]\n\t" \
  "s_bitcmp1_b64 s[78:79], s47\n\ts_cbranch_scc0 LSLOWN1_%=\nLBACKN1_%=:\n\t" \
  "v_mfma_f32_16x16x32_f16 a[240:243], v[60:63], v[80:83], a[240:243]\n\t" \
  "v_mfma_f32_16x16x32_f16 a[240:243], v[56:59], v[64:67], a[240:243]\n\t" \
  "LN2_%=:\n\t" \
  "s_cmp_eq_u32 s48, 2\n\t" \
  "s_cbranch_scc1 LC2_%=\n\t" \
  "s_waitcnt lgkmcnt(2)\n\t" \
  "v_mfma_f32_16x16x32_f16 a[0:3], v[32:35], v[96:99], a[0:3]\n\t" \
  "ds_read_b128 v[48:51], v25 offset:34816\n\t" \
  "v_mfma_f32_16x16x32_f16 a[0:3], v[36:39], v[112:115], a[0:3]\n\t" \
  "v_mfma_f32_16x16x32_f16 a[4:7], v[36:39], v[116:119], a[4:7]\n\t" \
  "buffer_load_dwordx4 v24, %[srdA], s42 offen offset:0 sc0 sc1 lds\n\t" \
  "v_mfma_f32_16x16x32_f16 a[4:7], v[32:35], v[100:103], a[4:7]\n\t" \
  "v_mfma_f32_16x16x32_f16 a[8:11], v[32:35], v[104:107], a[8:11]\n\t" \
  "ds_read_b128 v[52:55], v25 offset:51200\n\t" \
  "v_mfma_f32_16x16x32_f16 a[8:11], v[36:39], v[120:123], a[8:11]\n\t" \
  "v_mfma_f32_16x16x32_f16 a[12:15], v[36:39], v[124:127], a[12:15]\n\t" \
  "buffer_load_dwordx4 v24, %[srdA], s42 offen offset:1024 sc0 sc1 lds\n\t" \
  "v_mfma_f32_16x16x32_f16 a[12:15], v[32:35], v[108:111], a[12:15]\n\t" \
  "s_waitcnt lgkmcnt(2)\n\t" \
  "v_mfma_f32_16x16x32_f16 a[28:31], v[40:43], v[108:111], a[28:31]\n\t" \
  "ds_read_b128 v[56:59], v25 offset:35840\n\t" \
  "v_mfma_f32_16x16x32_f16 a[28:31], v[44:47], v[124:127], a[28:31]\n\t" \
  "v_mfma_f32_16x16x32_f16 a[24:27], v[44:47], v[120:123], a[24:27]\n\t" \
  "buffer_load_dwordx4 v24, %[srdA], s42 offen offset:2048 sc0 sc1 lds\n\t" \
  "v_mfma_f32_16x16x32_f16 a[24:27], v[40:43], v[104:107], a[24:27]\n\t" \
  "s_waitcnt vmcnt(21)\n\t" \
  "v_cvt_pk_f16_f32 v64, v128, v132\n\t" \
  "v_mfma_f32_16x16x32_f16 a[20:23], v[40:43], v[100:103], a[20:23]\n\t" \
  "ds_read_b128 v[60:63], v25 offset:52224\n\t" \
  "v_mfma_f32_16x16x32_f16 a[20:23], v[44:47], v[116:119], a[20:23]\n\t" \
  "v_cvt_pk_f16_f32 v68, v129, v133\n\t" \
  "v_mfma_f32_16x16x32_f16 a[16:19], v[44:47], v[112:115], a[16:19]\n\t" \
  "buffer_load_dwordx4 v24, %[srdA], s42 offen offset:3072 sc0 sc1 lds\n\t" \
  "v_mfma_f32_16x16x32_f16 a[16:19], v[40:43], v[96:99], a[16:19]\n\t" \
  "v_cvt_pk_f16_f32 v72, v130, v134\n\t" \
  "s_waitcnt lgkmcnt(2)\n\t" \
  "v_mfma_f32_16x16x32_f16 a[32:35], v[48:51], v[96:99], a[32:35]\n\t" \
  "ds_read_b128 v[32:35], v25 offset:36864\n\t" \
  "v_mfma_f32_16x16x32_f16 a[32:35], v[52:55], v[112:115], a[32:35]\n\t" \
  "v_cvt_pk_f16_f32 v76, v131, v135\n\t" \
  "v_mfma_f32_16x16x32_f16 a[36:39], v[52:55], v[116:119], a[36:39]\n\t" \
  "buffer_load_dwordx4 v[128:131], v16, %[srdW], s40 offen\n\t" \
  "v_mfma_f32_16x16x32_f16 a[36:39], v[48:51], v[100:103], a[36:39]\n\t" \
  "s_waitcnt vmcnt(21)\n\t" \
  "v_cvt_pk_f16_f32 v65, v136, v140\n\t" \
  "v_mfma_f32_16x16x32_f16 a[40:43], v[48:51], v[104:107], a[40:43]\n\t" \
  "ds_read_b128 v[36:39], v25 offset:53248\n\t" \
  "v_mfma_f32_16x16x32_f16 a[40:43], v[52:55], v[120:123], a[40:43]\n\t" \
  "v_cvt_pk_f16_f32 v69, v137, v141\n\t" \
  "v_mfma_f32_16x16x32_f16 a[44:47], v[52:55], v[124:127], a[44:47]\n\t" \
  "buffer_load_dwordx4 v[132:135], v17, %[srdW], s40 offen\n\t" \
  "v_mfma_f32_16x16x32_f16 a[44:47], v[48:51], v[108:111], a[44:47]\n\t" \
  "v_cvt_pk_f16_f32 v73, v138, v142\n\t" \
  "s_waitcnt lgkmcnt(2)\n\t" \
  "v_mfma_f32_16x16x32_f16 a[60:63], v[56:59], v[108:111], a[60:63]\n\t" \
  "ds_read_b128 v[40:43], v25 offset:37888\n\t" \
  "v_mfma_f32_16x16x32_f16 a[60:63], v[60:63], v[124:127], a[60:63]\n\t" \
  "v_cvt_pk_f16_f32 v77, v139, v143\n\t" \
  "v_mfma_f32_16x16x32_f16 a[56:59], v[60:63], v[120:123], a[56:59]\n\t" \
  "buffer_load_dwordx4 v[136:139], v18, %[srdW], s40 offen\n\t" \
  "v_mfma_f32_16x16x32_f16 a[56:59], v[56:59], v[104:107], a[56:59]\n\t" \
  "s_waitcnt vmcnt(21)\n\t" \
  "v_cvt_pk_f16_f32 v66, v144, v148\n\t" \
  "v_mfma_f32_16x16x32_f16 a[52:55], v[56:59], v[100:103], a[52:55]\n\t" \
  "ds_read_b128 v[44:47], v25 offset:54272\n\t" \
  "v_mfma_f32_16x16x32_f16 a[52:55], v[60:63], v[116:119], a[52:55]\n\t" \
  "v_cvt_pk_f16_f32 v70, v145, v149\n\t" \
  "v_mfma_f32_16x16x32_f16 a[48:51], v[60:63], v[112:115], a[48:51]\n\t" \
  "buffer_load_dwordx4 v[140:143], v19, %[srdW], s40 offen\n\t" \
  "v_mfma_f32_16x16x32_f16 a[48:51], v[56:59], v[96:99], a[48:51]\n\t" \
  "v_cvt_pk_f16_f32 v74, v146, v150\n\t" \
  "s_waitcnt lgkmcnt(2)\n\t" \
  "v_mfma_f32_16x16x32_f16 a[64:67], v[32:35], v[96:99], a[64:67]\n\t" \
  "ds_read_b128 v[48:51], v25 offset:38912\n\t" \
  "v_mfma_f32_16x16x32_f16 a[64:67], v[36:39], v[112:115], a[64:67]\n\t" \
  "v_cvt_pk_f16_f32 v78, v147, v151\n\t" \
  "v_mfma_f32_16x16x32_f16 a[68:71], v[36:39], v[116:119], a[68:71]\n\t" \
  "buffer_load_dwordx4 v[144:147], v20, %[srdW], s40 offen\n\t" \
  "v_mfma_f32_16x16x32_f16 a[68:71], v[32:35], v[100:103], a[68:71]\n\t" \
  "s_waitcnt vmcnt(21)\n\t" \
  "v_cvt_pk_f16_f32 v67, v152, v156\n\t" \
  "v_mfma_f32_16x16x32_f16 a[72:75], v[32:35], v[104:107], a[72:75]\n\t" \
  "ds_read_b128 v[52:55], v25 offset:55296\n\t" \
  "v_mfma_f32_16x16x32_f16 a[72:75], v[36:39], v[120:123], a[72:75]\n\t" \
  "v_cvt_pk_f16_f32 v71, v153, v157\n\t" \
  "v_mfma_f32_16x16x32_f16 a[76:79], v[36:39], v[124:127], a[76:79]\n\t" \
  "buffer_load_dwordx4 v[148:151], v21, %[srdW], s40 offen\n\t" \
  "v_mfma_f32_16x16x32_f16 a[76:79], v[32:35], v[108:111], a[76:79]\n\t" \
  "v_cvt_pk_f16_f32 v75, v154, v158\n\t" \
  "s_waitcnt lgkmcnt(2)\n\t" \
  "v_mfma_f32_16x16x32_f16 a[92:95], v[40:43], v[108:111], a[92:95]\n\t" \
  "ds_read_b128 v[56:59], v25 offset:39936\n\t" \
  "v_mfma_f32_16x16x32_f16 a[92:95], v[44:47], v[124:127], a[92:95]\n\t" \
  "v_cvt_pk_f16_f32 v79, v155, v159\n\t" \
  "v_mfma_f32_16x16x32_f16 a[88:91], v[44:47], v[120:123], a[88:91]\n\t" \
  "buffer_load_dwordx4 v[152:155], v22, %[srdW], s40 offen\n\t" \
  "v_mfma_f32_16x16x32_f16 a[88:91], v[40:43], v[104:107], a[88:91]\n\t" \
  "v_mfma_f32_16x16x32_f16 a[84:87], v[40:43], v[100:103], a[84:87]\n\t" \
  "ds_read_b128 v[60:63], v25 offset:56320\n\t" \
  "v_mfma_f32_16x16x32_f16 a[84:87], v[44:47], v[116:119], a[84:87]\n\t" \
  "v_mfma_f32_16x16x32_f16 a[80:83], v[44:47], v[112:115], a[80:83]\n\t" \
  "buffer_load_dwordx4 v[156:159], v23, %[srdW], s40 offen\n\t" \
  "v_mfma_f32_16x16x32_f16 a[80:83], v[40:43], v[96:99], a[80:83]\n\t" \
  "s_waitcnt lgkmcnt(2)\n\t" \
  "v_mfma_f32_16x16x32_f16 a[96:99], v[48:51], v[96:99], a[96:99]\n\t" \
  "ds_read_b128 v[32:35], v25 offset:40960\n\t" \
  "v_mfma_f32_16x16x32_f16 a[96:99], v[52:55], v[112:115], a[96:99]\n\t" \
  "v_cmp_le_u32 s[50:51], s59, v28\n\t" \
  "v_mfma_f32_16x16x32_f16 a[100:103], v[52:55], v[116:119], a[100:103]\n\t" \
  "v_cmp_le_u32 s[52:53], s59, v29\n\t" \
  "v_mfma_f32_16x16x32_f16 a[100:103], v[48:51], v[100:103], a[100:103]\n\t" \
  "s_lshr_b32 s47, s46, 16\n\t" \
  "v_mfma_f32_16x16x32_f16 a[104:107], v[48:51], v[104:107], a[104:107]\n\t" \
  "ds_read_b128 v[36:39], v25 offset:57344\n\t" \
  "v_mfma_f32_16x16x32_f16 a[104:107], v[52:55], v[120:123], a[104:107]\n\t" \
  "s_bitcmp1_b32 s47, 6\n\ts_cselect_b64 s[78:79], s[52:53], s[50:51]\n\t" \
  "v_mfma_f32_16x16x32_f16 a[108:111], v[52:55], v[124:127], a[108:111]\n\t" \
  "s_lshr_b32 s40, s47, 7\n\t" \
  "v_mfma_f32_16x16x32_f16 a[108:111], v[48:51], v[108:111], a[108:111]\n\t" \
  "s_lshl_b32 s42, s40, 16\n\t" \
  "s_waitcnt lgkmcnt(2)\n\t" \
  "v_mfma_f32_16x16x32_f16 a[124:127], v[56:59], v[108:111], a[124:127]\n\t" \
  "ds_read_b128 v[40:43], v25 offset:41984\n\t" \
  "v_mfma_f32_16x16x32_f16 a[124:127], v[60:63], v[124:127], a[124:127]\n\t" \
  "s_lshl_b32 s40, s40, 20\n\t" \
  "v_mfma_f32_16x16x32_f16 a[120:123], v[60:63], v[120:123], a[120:123]\n\t" \
  "s_add_u32 s43, s43, 0x4000\n\t" \
  "v_mfma_f32_16x16x32_f16 a[120:123], v[56:59], v[104:107], a[120:123]\n\t" \
  "s_and_b32 m0, s43, 0x1ffff\n\t" \
  "v_mfma_f32_16x16x32_f16 a[116:119], v[56:59], v[100:103], a[116:119]\n\t" \
  "ds_read_b128 v[44:47], v25 offset:58368\n\t" \
  "v_mfma_f32_16x16x32_f16 a[116:119], v[60:63], v[116:119], a[116:119]\n\t" \
  "s_bitcmp1_b64 s[78:79], s47\n\ts_cbranch_scc0 LSLOWN2_%=\nLBACKN2_%=:\n\t" \
  "v_mfma_f32_16x16x32_f16 a[112:115], v[60:63], v[112:115], a[112:115]\n\t" \
  "v_mfma_f32_16x16x32_f16 a[112:115], v[56:59], v[96:99], a[112:115]\n\t" \
  "LN3_%=:\n\t" \
  "s_cmp_eq_u32 s48, 3\n\t" \
  "s_cbranch_scc1 LC3_%=\n\t" \
  "s_waitcnt lgkmcnt(2)\n\t" \
  "v_mfma_f32_16x16x32_f16 a[128:131], v[32:35], v[96:99], a[128:131]\n\t" \
  "ds_read_b128 v[48:51], v25 offset:43008\n\t" \
  "v_mfma_f32_16x16x32_f16 a[128:131], v[36:39], v[112:115], a[128:131]\n\t" \
  "v_mfma_f32_16x16x32_f16 a[132:135], v[36:39], v[116:119], a[132:135]\n\t" \
  "buffer_load_dwordx4 v24, %[srdA], s42 offen offset:0 sc0 sc1 lds\n\t" \
  "v_mfma_f32_16x16x32_f16 a[132:135], v[32:35], v[100:103], a[132:135]\n\t" \
  "v_mfma_f32_16x16x32_f16 a[136:139], v[32:35], v[104:107], a[136:139]\n\t" \
  "ds_read_b128 v[52:55], v25 offset:59392\n\t" \
  "v_mfma_f32_16x16x32_f16 a[136:139], v[36:39], v[120:123], a[136:139]\n\t" \
  "v_mfma_f32_16x16x32_f16 a[140:143], v[36:39], v[124:127], a[140:143]\n\t" \
  "buffer_load_dwordx4 v24, %[srdA], s42 offen offset:1024 sc0 sc1 lds\n\t" \
  "v_mfma_f32_16x16x32_f16 a[140:143], v[32:35], v[108:111], a[140:143]\n\t" \
  "s_waitcnt lgkmcnt(2)\n\t" \
  "v_mfma_f32_16x16x32_f16 a[156:159], v[40:43], v[108:111], a[156:159]\n\t" \
  "ds_read_b128 v[56:59], v25 offset:44032\n\t" \
  "v_mfma_f32_16x16x32_f16 a[156:159], v[44:47], v[124:127], a[156:159]\n\t" \
  "v_mfma_f32_16x16x32_f16 a[152:155], v[44:47], v[120:123], a[152:155]\n\t" \
  "buffer_load_dwordx4 v24, %[srdA], s42 offen offset:2048 sc0 sc1 lds\n\t" \
  "v_mfma_f32_16x16x32_f16 a[152:155], v[40:43], v[104:107], a[152:155]\n\t" \
  "s_waitcnt vmcnt(21)\n\t" \
  "s_barrier\n\t" \
  "v_cvt_pk_f16_f32 v80, v160, v164\n\t" \
  "v_mfma_f32_16x16x32_f16 a[148:151], v[40:43], v[100:103], a[148:151]\n\t" \
  "ds_read_b128 v[60:63], v25 offset:60416\n\t" \
  "v_mfma_f32_16x16x32_f16 a[148:151], v[44:47], v[116:119], a[148:151]\n\t" \
  "v_cvt_pk_f16_f32 v84, v161, v165\n\t" \
  "v_mfma_f32_16x16x32_f16 a[144:147], v[44:47], v[112:115], a[144:147]\n\t" \
  "buffer_load_dwordx4 v24, %[srdA], s42 offen offset:3072 sc0 sc1 lds\n\t" \
  "v_mfma_f32_16x16x32_f16 a[144:147], v[40:43], v[96:99], a[144:147]\n\t" \
  "v_cvt_pk_f16_f32 v88, v162, v166\n\t" \
  "s_waitcnt lgkmcnt(2)\n\t" \
  "v_mfma_f32_16x16x32_f16 a[160:163], v[48:51], v[96:99], a[160:163]\n\t" \
  "ds_read_b128 v[32:35], v25 offset:45056\n\t" \
  "v_mfma_f32_16x16x32_f16 a[160:163], v[52:55], v[112:115], a[160:163]\n\t" \
  "v_cvt_pk_f16_f32 v92, v163, v167\n\t" \
  "v_mfma_f32_16x16x32_f16 a[164:167], v[52:55], v[116:119], a[164:167]\n\t" \
  "buffer_load_dwordx4 v[160:163], v16, %[srdW], s40 offen\n\t" \
  "v_mfma_f32_16x16x32_f16 a[164:167], v[48:51], v[100:103], a[164:167]\n\t" \
  "s_waitcnt vmcnt(21)\n\t" \
  "v_cvt_pk_f16_f32 v81, v168, v172\n\t" \
  "v_mfma_f32_16x16x32_f16 a[168:171], v[48:51], v[104:107], a[168:171]\n\t" \
  "ds_read_b128 v[36:39], v25 offset:61440\n\t" \
  "v_mfma_f32_16x16x32_f16 a[168:171], v[52:55], v[120:123], a[168:171]\n\t" \
  "v_cvt_pk_f16_f32 v85, v169, v173\n\t" \
  "v_mfma_f32_16x16x32_f16 a[172:175], v[52:55], v[124:127], a[172:175]\n\t" \
  "buffer_load_dwordx4 v[164:167], v17, %[srdW], s40 offen\n\t" \
  "v_mfma_f32_16x16x32_f16 a[172:175], v[48:51], v[108:111], a[172:175]\n\t" \
  "v_cvt_pk_f16_f32 v89, v170, v174\n\t" \
  "s_waitcnt lgkmcnt(2)\n\t" \
  "v_mfma_f32_16x16x32_f16 a[188:191], v[56:59], v[108:111], a[188:191]\n\t" \
  "ds_read_b128 v[40:43], v25 offset:46080\n\t" \
  "v_mfma_f32_16x16x32_f16 a[188:191], v[60:63], v[124:127], a[188:191]\n\t" \
  "v_cvt_pk_f16_f32 v93, v171, v175\n\t" \
  "v_mfma_f32_16x16x32_f16 a[184:187], v[60:63], v[120:123], a[184:187]\n\t" \
  "buffer_load_dwordx4 v[168:171], v18, %[srdW], s40 offen\n\t" \
  "v_mfma_f32_16x16x32_f16 a[184:187], v[56:59], v[104:107], a[184:187]\n\t" \
  "s_waitcnt vmcnt(21)\n\t" \
  "v_cvt_pk_f16_f32 v82, v176, v180\n\t" \
  "v_mfma_f32_16x16x32_f16 a[180:183], v[56:59], v[100:103], a[180:183]\n\t" \
  "ds_read_b128 v[44:47], v25 offset:62464\n\t" \
  "v_mfma_f32_16x16x32_f16 a[180:183], v[60:63], v[116:119], a[180:183]\n\t" \
  "v_cvt_pk_f16_f32 v86, v177, v181\n\t" \
  "v_mfma_f32_16x16x32_f16 a[176:179], v[60:63], v[112:115], a[176:179]\n\t" \
  "buffer_load_dwordx4 v[172:175], v19, %[srdW], s40 offen\n\t" \
  "v_mfma_f32_16x16x32_f16 a[176:179], v[56:59], v[96:99], a[176:179]\n\t" \
  "v_cvt_pk_f16_f32 v90, v178, v182\n\t" \
  "s_waitcnt lgkmcnt(2)\n\t" \
  "v_mfma_f32_16x16x32_f16 a[192:195], v[32:35], v[96:99], a[192:195]\n\t" \
  "ds_read_b128 v[48:51], v25 offset:47104\n\t" \
  "v_mfma_f32_16x16x32_f16 a[192:195], v[36:39], v[112:115], a[192:195]\n\t" \
  "v_cvt_pk_f16_f32 v94, v179, v183\n\t" \
  "v_mfma_f32_16x16x32_f16 a[196:199], v[36:39], v[116:119], a[196:199]\n\t" \
  "buffer_load_dwordx4 v[176:179], v20, %[srdW], s40 offen\n\t" \
  "v_mfma_f32_16x16x32_f16 a[196:199], v[32:35], v[100:103], a[196:199]\n\t" \
  "s_waitcnt vmcnt(21)\n\t" \
  "v_cvt_pk_f16_f32 v83, v184, v188\n\t" \
  "v_mfma_f32_16x16x32_f16 a[200:203], v[32:35], v[104:107], a[200:203]\n\t" \
  "ds_read_b128 v[52:55], v25 offset:63488\n\t" \
  "v_mfma_f32_16x16x32_f16 a[200:203], v[36:39], v[120:123], a[200:203]\n\t" \
  "v_cvt_pk_f16_f32 v87, v185, v189\n\t" \
  "v_mfma_f32_16x16x32_f16 a[204:207], v[36:39], v[124:127], a[204:207]\n\t" \
  "buffer_load_dwordx4 v[180:183], v21, %[srdW], s40 offen\n\t" \
  "v_mfma_f32_16x16x32_f16 a[204:207], v[32:35], v[108:111], a[204:207]\n\t" \
  "v_cvt_pk_f16_f32 v91, v186, v190\n\t" \
  "s_waitcnt lgkmcnt(2)\n\t" \
  "v_mfma_f32_16x16x32_f16 a[220:223], v[40:43], v[108:111], a[220:223]\n\t" \
  "ds_read_b128 v[56:59], v25 offset:48128\n\t" \
  "v_mfma_f32_16x16x32_f16 a[220:223], v[44:47], v[124:127], a[220:223]\n\t" \
  "v_cvt_pk_f16_f32 v95, v187, v191\n\t" \
  "v_mfma_f32_16x16x32_f16 a[216:219], v[44:47], v[120:123], a[216:219]\n\t" \
  "buffer_load_dwordx4 v[184:187], v22, %[srdW], s40 offen\n\t" \
  "v_mfma_f32_16x16x32_f16 a[216:219], v[40:43], v[104:107], a[216:219]\n\t" \
  "v_mfma_f32_16x16x32_f16 a[212:215], v[40:43], v[100:103], a[212:215]\n\t" \
  "ds_read_b128 v[60:63], v25 offset:64512\n\t" \
  "v_mfma_f32_16x16x32_f16 a[212:215], v[44:47], v[116:119], a[212:215]\n\t" \
  "v_readlane_b32 s46, %[vTab], s44\n\t" \
  "v_mfma_f32_16x16x32_f16 a[208:211], v[44:47], v[112:115], a[208:211]\n\t" \
  "buffer_load_dwordx4 v[188:191], v23, %[srdW], s40 offen\n\t" \
  "v_mfma_f32_16x16x32_f16 a[208:211], v[40:43], v[96:99], a[208:211]\n\t" \
  "v_xor_b32 v25, 0x10000, v25\n\t" \
  "s_add_u32 s44, s44, 1\n\t" \
  "s_waitcnt lgkmcnt(2)\n\t" \
  "v_mfma_f32_16x16x32_f16 a[224:227], v[48:51], v[96:99], a[224:227]\n\t" \
  "ds_read_b128 v[32:35], v25 offset:0\n\t" \
  "v_mfma_f32_16x16x32_f16 a[224:227], v[52:55], v[112:115], a[224:227]\n\t" \
  "v_cmp_le_u32 s[50:51], s59, v28\n\t" \
  "v_mfma_f32_16x16x32_f16 a[228:231], v[52:55], v[116:119], a[228:231]\n\t" \
  "v_cmp_le_u32 s[52:53], s59, v29\n\t" \
  "v_mfma_f32_16x16x32_f16 a[228:231], v[48:51], v[100:103], a[228:231]\n\t" \
  "s_and_b32 s47, s46, 0xffff\n\t" \
  "v_mfma_f32_16x16x32_f16 a[232:235], v[48:51], v[104:107], a[232:235]\n\t" \
  "ds_read_b128 v[36:39], v25 offset:16384\n\t" \
  "v_mfma_f32_16x16x32_f16 a[232:235], v[52:55], v[120:123], a[232:235]\n\t" \
  "s_bitcmp1_b32 s47, 6\n\ts_cselect_b64 s[78:79], s[52:53], s[50:51]\n\t" \
  "v_mfma_f32_16x16x32_f16 a[236:239], v[52:55], v[124:127], a[236:239]\n\t" \
  "s_lshr_b32 s40, s47, 7\n\t" \
  "v_mfma_f32_16x16x32_f16 a[236:239], v[48:51], v[108:111], a[236:239]\n\t" \
  "s_lshl_b32 s42, s40, 16\n\t" \
  "s_waitcnt lgkmcnt(2)\n\t" \
  "v_mfma_f32_16x16x32_f16 a[252:255], v[56:59], v[108:111], a[252:255]\n\t" \
  "ds_read_b128 v[40:43], v25 offset:1024\n\t" \
  "v_mfma_f32_16x16x32_f16 a[252:255], v[60:63], v[124:127], a[252:255]\n\t" \
  "s_lshl_b32 s40, s40, 20\n\t" \
  "v_mfma_f32_16x16x32_f16 a[248:251], v[60:63], v[120:123], a[248:251]\n\t" \
  "s_add_u32 s43, s43, 0x4000\n\t" \
  "v_mfma_f32_16x16x32_f16 a[248:251], v[56:59], v[104:107], a[248:251]\n\t" \
  "s_and_b32 m0, s43, 0x1ffff\n\t" \
  "v_mfma_f32_16x16x32_f16 a[244:247], v[56:59], v[100:103], a[244:247]\n\t" \
  "ds_read_b128 v[44:47], v25 offset:17408\n\t" \
  "v_mfma_f32_16x16x32_f16 a[244:247], v[60:63], v[116:119], a[244:247]\n\t" \
  "s_bitcmp1_b64 s[78:79], s47\n\ts_cbranch_scc0 LSLOWN3_%=\nLBACKN3_%=:\n\t" \
  "v_mfma_f32_16x16x32_f16 a[240:243], v[60:63], v[112:115], a[240:243]\n\t" \
  "v_mfma_f32_16x16x32_f16 a[240:243], v[56:59], v[96:99], a[240:243]\n\t" \
  "LN4_%=:\n\t" \
  "s_sub_u32 s45, s45, 1\n\t" \
  "s_cmp_lg_u32 s45, 0\n\t" \
  "s_cbranch_scc1 LN0_%=\n\t" \
  "s_branch LEND_%=\n\t" \
  ".p2align 6\n\t" \
  "LC0_%=:\n\t" \
  "s_waitcnt lgkmcnt(2)\n\t" \
  "v_mfma_f32_16x16x32_f16 a[0:3], v[32:35], v[64:67], a[0:3]\n\t" \
  "ds_read_b128 v[48:51], v25 offset:2048\n\t" \
  "v_readlane_b32 s54, %[vDlo], s49\n\t" \
  "v_mfma_f32_16x16x32_f16 a[0:3], v[36:39], v[80:83], a[0:3]\n\t" \
  "v_readlane_b32 s55, %[vDhi], s49\n\t" \
  "v_readlane_b32 s56, %[vGlo], s49\n\t" \
  "v_mfma_f32_16x16x32_f16 a[4:7], v[36:39], v[84:87], a[4:7]\n\t" \
  "buffer_load_dwordx4 v24, %[srdA], s42 offen offset:0 sc0 sc1 lds\n\t" \
  "v_mfma_f32_16x16x32_f16 a[4:7], v[32:35], v[68:71], a[4:7]\n\t" \
  "v_readlane_b32 s57, %[vGhi], s49\n\t" \
  "v_readlane_b32 s58, %[vXo], s49\n\t" \
  "v_mfma_f32_16x16x32_f16 a[8:11], v[32:35], v[72:75], a[8:11]\n\t" \
  "ds_read_b128 v[52:55], v25 offset:18432\n\t" \
  "v_mfma_f32_16x16x32_f16 a[8:11], v[36:39], v[88:91], a[8:11]\n\t" \
  "s_add_u32 s49, s49, 1\n\t" \
  "s_cmp_lg_u32 s55, 0\n\ts_cselect_b64 s[64:65], -1, 0\n\t" \
  "v_mfma_f32_16x16x32_f16 a[12:15], v[36:39], v[92:95], a[12:15]\n\t" \
  "buffer_load_dwordx4 v24, %[srdA], s42 offen offset:1024 sc0 sc1 lds\n\t" \
  "v_mfma_f32_16x16x32_f16 a[12:15], v[32:35], v[76:79], a[12:15]\n\t" \
  "s_waitcnt lgkmcnt(2)\n\t" \
  "v_mfma_f32_16x16x32_f16 a[28:31], v[40:43], v[76:79], a[28:31]\n\t" \
  "ds_read_b128 v[56:59], v25 offset:3072\n\t" \
  "v_mfma_f32_16x16x32_f16 a[28:31], v[44:47], v[92:95], a[28:31]\n\t" \
  "s_cmp_lg_u32 s57, 0\n\ts_cselect_b64 s[66:67], 1, 0\n\t" \
  "s_cmp_ge_u32 s49, %[sZN]\n\ts_cselect_b32 s48, 99, s48\n\t" \
  "v_mfma_f32_16x16x32_f16 a[24:27], v[44:47], v[88:91], a[24:27]\n\t" \
  "buffer_load_dwordx4 v24, %[srdA], s42 offen offset:2048 sc0 sc1 lds\n\t" \
  "v_mfma_f32_16x16x32_f16 a[24:27], v[40:43], v[72:75], a[24:27]\n\t" \
  "v_cvt_pk_f16_f32 v200, v192, v193\n\t" \
  "v_mfma_f32_16x16x32_f16 a[20:23], v[40:43], v[68:71], a[20:23]\n\t" \
  "ds_read_b128 v[60:63], v25 offset:19456\n\t" \
  "v_mfma_f32_16x16x32_f16 a[20:23], v[44:47], v[84:87], a[20:23]\n\t" \
  "v_cvt_pk_f16_f32 v201, v194, v195\n\t" \
  "v_cvt_pk_f16_f32 v202, v196, v197\n\t" \
  "v_mfma_f32_16x16x32_f16 a[16:19], v[44:47], v[80:83], a[16:19]\n\t" \
  "buffer_load_dwordx4 v24, %[srdA], s42 offen offset:3072 sc0 sc1 lds\n\t" \
  "v_mfma_f32_16x16x32_f16 a[16:19], v[40:43], v[64:67], a[16:19]\n\t" \
  "v_cvt_pk_f16_f32 v203, v198, v199\n\t" \
  "s_waitcnt lgkmcnt(2)\n\t" \
  "v_mfma_f32_16x16x32_f16 a[32:35], v[48:51], v[64:67], a[32:35]\n\t" \
  "ds_read_b128 v[32:35], v25 offset:4096\n\t" \
  "v_mfma_f32_16x16x32_f16 a[32:35], v[52:55], v[80:83], a[32:35]\n\t" \
  "s_mov_b64 exec, s[66:67]\n\tglobal_store_dword v31, v27, s[56:57] sc0 sc1\n\ts_mov_b64 exec, -1\n\t" \
  "v_mfma_f32_16x16x32_f16 a[36:39], v[52:55], v[84:87], a[36:39]\n\t" \
  "s_mov_b64 exec, s[64:65]\n\tglobal_store_dwordx4 v26, v[200:203], s[54:55] sc0 sc1\n\ts_mov_b64 exec, -1\n\t" \
  "v_mfma_f32_16x16x32_f16 a[36:39], v[48:51], v[68:71], a[36:39]\n\t" \
  "v_mfma_f32_16x16x32_f16 a[40:43], v[48:51], v[72:75], a[40:43]\n\t" \
  "ds_read_b128 v[36:39], v25 offset:20480\n\t" \
  "v_mfma_f32_16x16x32_f16 a[40:43], v[52:55], v[88:91], a[40:43]\n\t" \
  "v_cmp_le_u32 s[50:51], s59, v28\n\t" \
  "v_mfma_f32_16x16x32_f16 a[44:47], v[52:55], v[92:95], a[44:47]\n\t" \
  "buffer_load_dwordx4 v[192:195], %[vXa], %[srdX], s58 offen\n\t" \
  "v_mfma_f32_16x16x32_f16 a[44:47], v[48:51], v[76:79], a[44:47]\n\t" \
  "s_waitcnt vmcnt(23)\n\t" \
  "v_cvt_pk_f16_f32 v96, v128, v132\n\t" \
  "s_waitcnt lgkmcnt(2)\n\t" \
  "v_mfma_f32_16x16x32_f16 a[60:63], v[56:59], v[76:79], a[60:63]\n\t" \
  "ds_read_b128 v[40:43], v25 offset:5120\n\t" \
  "v_mfma_f32_16x16x32_f16 a[60:63], v[60:63], v[92:95], a[60:63]\n\t" \
  "v_cvt_pk_f16_f32 v100, v129, v133\n\t" \
  "v_mfma_f32_16x16x32_f16 a[56:59], v[60:63], v[88:91], a[56:59]\n\t" \
  "buffer_load_dwordx4 v[196:199], %[vXa], %[srdX], s58 offen offset:64\n\t" \
  "v_mfma_f32_16x16x32_f16 a[56:59], v[56:59], v[72:75], a[56:59]\n\t" \
  "v_cvt_pk_f16_f32 v104, v130, v134\n\t" \
  "v_mfma_f32_16x16x32_f16 a[52:55], v[56:59], v[68:71], a[52:55]\n\t" \
  "ds_read_b128 v[44:47], v25 offset:21504\n\t" \
  "v_mfma_f32_16x16x32_f16 a[52:55], v[60:63], v[84:87], a[52:55]\n\t" \
  "v_cvt_pk_f16_f32 v108, v131, v135\n\t" \
  "v_mfma_f32_16x16x32_f16 a[48:51], v[60:63], v[80:83], a[48:51]\n\t" \
  "buffer_load_dwordx4 v[128:131], v16, %[srdW], s40 offen\n\t" \
  "v_mfma_f32_16x16x32_f16 a[48:51], v[56:59], v[64:67], a[48:51]\n\t" \
  "s_waitcnt vmcnt(23)\n\t" \
  "v_cvt_pk_f16_f32 v97, v136, v140\n\t" \
  "s_waitcnt lgkmcnt(2)\n\t" \
  "v_mfma_f32_16x16x32_f16 a[64:67], v[32:35], v[64:67], a[64:67]\n\t" \
  "ds_read_b128 v[48:51], v25 offset:6144\n\t" \
  "v_mfma_f32_16x16x32_f16 a[64:67], v[36:39], v[80:83], a[64:67]\n\t" \
  "v_cvt_pk_f16_f32 v101, v137, v141\n\t" \
  "v_mfma_f32_16x16x32_f16 a[68:71], v[36:39], v[84:87], a[68:71]\n\t" \
  "buffer_load_dwordx4 v[132:135], v17, %[srdW], s40 offen\n\t" \
  "v_mfma_f32_16x16x32_f16 a[68:71], v[32:35], v[68:71], a[68:71]\n\t" \
  "v_cvt_pk_f16_f32 v105, v138, v142\n\t" \
  "v_mfma_f32_16x16x32_f16 a[72:75], v[32:35], v[72:75], a[72:75]\n\t" \
  "ds_read_b128 v[52:55], v25 offset:22528\n\t" \
  "v_mfma_f32_16x16x32_f16 a[72:75], v[36:39], v[88:91], a[72:75]\n\t" \
  "v_cvt_pk_f16_f32 v109, v139, v143\n\t" \
  "v_mfma_f32_16x16x32_f16 a[76:79], v[36:39], v[92:95], a[76:79]\n\t" \
  "buffer_load_dwordx4 v[136:139], v18, %[srdW], s40 offen\n\t" \
  "v_mfma_f32_16x16x32_f16 a[76:79], v[32:35], v[76:79], a[76:79]\n\t" \
  "s_waitcnt vmcnt(23)\n\t" \
  "v_cvt_pk_f16_f32 v98, v144, v148\n\t" \
  "s_waitcnt lgkmcnt(2)\n\t" \
  "v_mfma_f32_16x16x32_f16 a[92:95], v[40:43], v[76:79], a[92:95]\n\t" \
  "ds_read_b128 v[56:59], v25 offset:7168\n\t" \
  "v_mfma_f32_16x16x32_f16 a[92:95], v[44:47], v[92:95], a[92:95]\n\t" \
  "v_cvt_pk_f16_f32 v102, v145, v149\n\t" \
  "v_mfma_f32_16x16x32_f16 a[88:91], v[44:47], v[88:91], a[88:91]\n\t" \
  "buffer_load_dwordx4 v[140:143], v19, %[srdW], s40 offen\n\t" \
  "v_mfma_f32_16x16x32_f16 a[88:91], v[40:43], v[72:75], a[88:91]\n\t" \
  "v_cvt_pk_f16_f32 v106, v146, v150\n\t" \
  "v_mfma_f32_16x16x32_f16 a[84:87], v[40:43], v[68:71], a[84:87]\n\t" \
  "ds_read_b128 v[60:63], v25 offset:23552\n\t" \
  "v_mfma_f32_16x16x32_f16 a[84:87], v[44:47], v[84:87], a[84:87]\n\t" \
  "v_cvt_pk_f16_f32 v110, v147, v151\n\t" \
  "v_mfma_f32_16x16x32_f16 a[80:83], v[44:47], v[80:83], a[80:83]\n\t" \
  "buffer_load_dwordx4 v[144:147], v20, %[srdW], s40 offen\n\t" \
  "v_mfma_f32_16x16x32_f16 a[80:83], v[40:43], v[64:67], a[80:83]\n\t" \
  "s_waitcnt vmcnt(23)\n\t" \
  "v_cvt_pk_f16_f32 v99, v152, v156\n\t" \
  "s_waitcnt lgkmcnt(2)\n\t" \
  "v_mfma_f32_16x16x32_f16 a[96:99], v[48:51], v[64:67], a[96:99]\n\t" \
  "ds_read_b128 v[32:35], v25 offset:8192\n\t" \
  "v_mfma_f32_16x16x32_f16 a[96:99], v[52:55], v[80:83], a[96:99]\n\t" \
  "v_cvt_pk_f16_f32 v103, v153, v157\n\t" \
  "v_mfma_f32_16x16x32_f16 a[100:103], v[52:55], v[84:87], a[100:103]\n\t" \
  "buffer_load_dwordx4 v[148:151], v21, %[srdW], s40 offen\n\t" \
  "v_mfma_f32_16x16x32_f16 a[100:103], v[48:51], v[68:71], a[100:103]\n\t" \
  "v_cvt_pk_f16_f32 v107, v154, v158\n\t" \
  "v_mfma_f32_16x16x32_f16 a[104:107], v[48:51], v[72:75], a[104:107]\n\t" \
  "ds_read_b128 v[36:39], v25 offset:24576\n\t" \
  "v_mfma_f32_16x16x32_f16 a[104:107], v[52:55], v[88:91], a[104:107]\n\t" \
  "v_cvt_pk_f16_f32 v111, v155, v159\n\t" \
  "v_mfma_f32_16x16x32_f16 a[108:111], v[52:55], v[92:95], a[108:111]\n\t" \
  "buffer_load_dwordx4 v[152:155], v22, %[srdW], s40 offen\n\t" \
  "v_mfma_f32_16x16x32_f16 a[108:111], v[48:51], v[76:79], a[108:111]\n\t" \
  "v_cmp_le_u32 s[52:53], s59, v29\n\t" \
  "s_waitcnt lgkmcnt(2)\n\t" \
  "v_mfma_f32_16x16x32_f16 a[124:127], v[56:59], v[76:79], a[124:127]\n\t" \
  "ds_read_b128 v[40:43], v25 offset:9216\n\t" \
  "v_mfma_f32_16x16x32_f16 a[124:127], v[60:63], v[92:95], a[124:127]\n\t" \
  "s_lshr_b32 s47, s46, 16\n\t" \
  "s_bitcmp1_b32 s47, 6\n\ts_cselect_b64 s[78:79], s[52:53], s[50:51]\n\t" \
  "v_mfma_f32_16x16x32_f16 a[120:123], v[60:63], v[88:91], a[120:123]\n\t" \
  "buffer_load_dwordx4 v[156:159], v23, %[srdW], s40 offen\n\t" \
  "v_mfma_f32_16x16x32_f16 a[120:123], v[56:59], v[72:75], a[120:123]\n\t" \
  "s_lshr_b32 s40, s47, 7\n\t" \
  "s_lshl_b32 s42, s40, 16\n\t" \
  "v_mfma_f32_16x16x32_f16 a[116:119], v[56:59], v[68:71], a[116:119]\n\t" \
  "ds_read_b128 v[44:47], v25 offset:25600\n\t" \
  "s_lshl_b32 s40, s40, 20\n\t" \
  "v_mfma_f32_16x16x32_f16 a[116:119], v[60:63], v[84:87], a[116:119]\n\t" \
  "s_add_u32 s43, s43, 0x4000\n\t" \
  "s_and_b32 m0, s43, 0x1ffff\n\t" \
  "v_mfma_f32_16x16x32_f16 a[112:115], v[60:63], v[80:83], a[112:115]\n\t" \
  "buffer_load_dwordx2 v[28:29], %[vFo], %[srdC], 0 offen sc0 sc1\n\t" \
  "v_mfma_f32_16x16x32_f16 a[112:115], v[56:59], v[64:67], a[112:115]\n\t" \
  "s_bitcmp1_b64 s[78:79], s47\n\ts_cbranch_scc0 LSLOWC0_%=\nLBACKC0_%=:\n\t" \
  "s_branch LN1_%=\n\t" \
  ".p2align 6\n\t" \
  "LC1_%=:\n\t" \
  "s_waitcnt lgkmcnt(2)\n\t" \
  "v_mfma_f32_16x16x32_f16 a[128:131], v[32:35], v[64:67], a[128:131]\n\t" \
  "ds_read_b128 v[48:51], v25 offset:10240\n\t" \
  "v_readlane_b32 s54, %[vDlo], s49\n\t" \
  "v_mfma_f32_16x16x32_f16 a[128:131], v[36:39], v[80:83], a[128:131]\n\t" \
  "v_readlane_b32 s55, %[vDhi], s49\n\t" \
  "v_readlane_b32 s56, %[vGlo], s49\n\t" \
  "v_mfma_f32_16x16x32_f16 a[132:135], v[36:39], v[84:87], a[132:135]\n\t" \
  "buffer_load_dwordx4 v24, %[srdA], s42 offen offset:0 sc0 sc1 lds\n\t" \
  "v_mfma_f32_16x16x32_f16 a[132:135], v[32:35], v[68:71], a[132:135]\n\t" \
  "v_readlane_b32 s57, %[vGhi], s49\n\t" \
  "v_readlane_b32 s58, %[vXo], s49\n\t" \
  "v_mfma_f32_16x16x32_f16 a[136:139], v[32:35], v[72:75], a[136:139]\n\t" \
  "ds_read_b128 v[52:55], v25 offset:26624\n\t" \
  "v_mfma_f32_16x16x32_f16 a[136:139], v[36:39], v[88:91], a[136:139]\n\t" \
  "s_add_u32 s49, s49, 1\n\t" \
  "s_cmp_lg_u32 s55, 0\n\ts_cselect_b64 s[64:65], -1, 0\n\t" \
  "v_mfma_f32_16x16x32_f16 a[140:143], v[36:39], v[92:95], a[140:143]\n\t" \
  "buffer_load_dwordx4 v24, %[srdA], s42 offen offset:1024 sc0 sc1 lds\n\t" \
  "v_mfma_f32_16x16x32_f16 a[140:143], v[32:35], v[76:79], a[140:143]\n\t" \
  "s_waitcnt lgkmcnt(2)\n\t" \
  "v_mfma_f32_16x16x32_f16 a[156:159], v[40:43], v[76:79], a[156:159]\n\t" \
  "ds_read_b128 v[56:59], v25 offset:11264\n\t" \
  "v_mfma_f32_16x16x32_f16 a[156:159], v[44:47], v[92:95], a[156:159]\n\t" \
  "s_cmp_lg_u32 s57, 0\n\ts_cselect_b64 s[66:67], 1, 0\n\t" \
  "s_cmp_ge_u32 s49, %[sZN]\n\ts_cselect_b32 s48, 99, s48\n\t" \
  "v_mfma_f32_16x16x32_f16 a[152:155], v[44:47], v[88:91], a[152:155]\n\t" \
  "buffer_load_dwordx4 v24, %[srdA], s42 offen offset:2048 sc0 sc1 lds\n\t" \
  "v_mfma_f32_16x16x32_f16 a[152:155], v[40:43], v[72:75], a[152:155]\n\t" \
  "s_waitcnt vmcnt(21)\n\t" \
  "s_barrier\n\t" \
  "v_cvt_pk_f16_f32 v200, v192, v193\n\t" \
  "v_mfma_f32_16x16x32_f16 a[148:151], v[40:43], v[68:71], a[148:151]\n\t" \
  "ds_read_b128 v[60:63], v25 offset:27648\n\t" \
  "v_mfma_f32_16x16x32_f16 a[148:151], v[44:47], v[84:87], a[148:151]\n\t" \
  "v_cvt_pk_f16_f32 v201, v194, v195\n\t" \
  "v_cvt_pk_f16_f32 v202, v196, v197\n\t" \
  "v_mfma_f32_16x16x32_f16 a[144:147], v[44:47], v[80:83], a[144:147]\n\t" \
  "buffer_load_dwordx4 v24, %[srdA], s42 offen offset:3072 sc0 sc1 lds\n\t" \
  "v_mfma_f32_16x16x32_f16 a[144:147], v[40:43], v[64:67], a[144:147]\n\t" \
  "v_cvt_pk_f16_f32 v203, v198, v199\n\t" \
  "s_waitcnt lgkmcnt(2)\n\t" \
  "v_mfma_f32_16x16x32_f16 a[160:163], v[48:51], v[64:67], a[160:163]\n\t" \
  "ds_read_b128 v[32:35], v25 offset:12288\n\t" \
  "v_mfma_f32_16x16x32_f16 a[160:163], v[52:55], v[80:83], a[160:163]\n\t" \
  "s_mov_b64 exec, s[66:67]\n\tglobal_store_dword v31, v27, s[56:57] sc0 sc1\n\ts_mov_b64 exec, -1\n\t" \
  "v_mfma_f32_16x16x32_f16 a[164:167], v[52:55], v[84:87], a[164:167]\n\t" \
  "s_mov_b64 exec, s[64:65]\n\tglobal_store_dwordx4 v26, v[200:203], s[54:55] sc0 sc1\n\ts_mov_b64 exec, -1\n\t" \
  "v_mfma_f32_16x16x32_f16 a[164:167], v[48:51], v[68:71], a[164:167]\n\t" \
  "v_readlane_b32 s46, %[vTab], s44\n\t" \
  "v_mfma_f32_16x16x32_f16 a[168:171], v[48:51], v[72:75], a[168:171]\n\t" \
  "ds_read_b128 v[36:39], v25 offset:28672\n\t" \
  "v_mfma_f32_16x16x32_f16 a[168:171], v[52:55], v[88:91], a[168:171]\n\t" \
  "s_add_u32 s44, s44, 1\n\t" \
  "v_cmp_le_u32 s[50:51], s59, v28\n\t" \
  "v_mfma_f32_16x16x32_f16 a[172:175], v[52:55], v[92:95], a[172:175]\n\t" \
  "buffer_load_dwordx4 v[192:195], %[vXa], %[srdX], s58 offen\n\t" \
  "v_mfma_f32_16x16x32_f16 a[172:175], v[48:51], v[76:79], a[172:175]\n\t" \
  "v_cvt_pk_f16_f32 v112, v160, v164\n\t" \
  "s_waitcnt lgkmcnt(2)\n\t" \
  "v_mfma_f32_16x16x32_f16 a[188:191], v[56:59], v[76:79], a[188:191]\n\t" \
  "ds_read_b128 v[40:43], v25 offset:13312\n\t" \
  "v_mfma_f32_16x16x32_f16 a[188:191], v[60:63], v[92:95], a[188:191]\n\t" \
  "v_cvt_pk_f16_f32 v116, v161, v165\n\t" \
  "v_mfma_f32_16x16x32_f16 a[184:187], v[60:63], v[88:91], a[184:187]\n\t" \
  "buffer_load_dwordx4 v[196:199], %[vXa], %[srdX], s58 offen offset:64\n\t" \
  "v_mfma_f32_16x16x32_f16 a[184:187], v[56:59], v[72:75], a[184:187]\n\t" \
  "v_cvt_pk_f16_f32 v120, v162, v166\n\t" \
  "v_mfma_f32_16x16x32_f16 a[180:183], v[56:59], v[68:71], a[180:183]\n\t" \
  "ds_read_b128 v[44:47], v25 offset:29696\n\t" \
  "v_mfma_f32_16x16x32_f16 a[180:183], v[60:63], v[84:87], a[180:183]\n\t" \
  "v_cvt_pk_f16_f32 v124, v163, v167\n\t" \
  "v_mfma_f32_16x16x32_f16 a[176:179], v[60:63], v[80:83], a[176:179]\n\t" \
  "buffer_load_dwordx4 v[160:163], v16, %[srdW], s40 offen\n\t" \
  "v_mfma_f32_16x16x32_f16 a[176:179], v[56:59], v[64:67], a[176:179]\n\t" \
  "s_waitcnt vmcnt(23)\n\t" \
  "v_cvt_pk_f16_f32 v113, v168, v172\n\t" \
  "s_waitcnt lgkmcnt(2)\n\t" \
  "v_mfma_f32_16x16x32_f16 a[192:195], v[32:35], v[64:67], a[192:195]\n\t" \
  "ds_read_b128 v[48:51], v25 offset:14336\n\t" \
  "v_mfma_f32_16x16x32_f16 a[192:195], v[36:39], v[80:83], a[192:195]\n\t" \
  "v_cvt_pk_f16_f32 v117, v169, v173\n\t" \
  "v_mfma_f32_16x16x32_f16 a[196:199], v[36:39], v[84:87], a[196:199]\n\t" \
  "buffer_load_dwordx4 v[164:167], v17, %[srdW], s40 offen\n\t" \
  "v_mfma_f32_16x16x32_f16 a[196:199], v[32:35], v[68:71], a[196:199]\n\t" \
  "v_cvt_pk_f16_f32 v121, v170, v174\n\t" \
  "v_mfma_f32_16x16x32_f16 a[200:203], v[32:35], v[72:75], a[200:203]\n\t" \
  "ds_read_b128 v[52:55], v25 offset:30720\n\t" \
  "v_mfma_f32_16x16x32_f16 a[200:203], v[36:39], v[88:91], a[200:203]\n\t" \
  "v_cvt_pk_f16_f32 v125, v171, v175\n\t" \
  "v_mfma_f32_16x16x32_f16 a[204:207], v[36:39], v[92:95], a[204:207]\n\t" \
  "buffer_load_dwordx4 v[168:171], v18, %[srdW], s40 offen\n\t" \
  "v_mfma_f32_16x16x32_f16 a[204:207], v[32:35], v[76:79], a[204:207]\n\t" \
  "s_waitcnt vmcnt(23)\n\t" \
  "v_cvt_pk_f16_f32 v114, v176, v180\n\t" \
  "s_waitcnt lgkmcnt(2)\n\t" \
  "v_mfma_f32_16x16x32_f16 a[220:223], v[40:43], v[76:79], a[220:223]\n\t" \
  "ds_read_b128 v[56:59], v25 offset:15360\n\t" \
  "v_mfma_f32_16x16x32_f16 a[220:223], v[44:47], v[92:95], a[220:223]\n\t" \
  "v_cvt_pk_f16_f32 v118, v177, v181\n\t" \
  "v_mfma_f32_16x16x32_f16 a[216:219], v[44:47], v[88:91], a[216:219]\n\t" \
  "buffer_load_dwordx4 v[172:175], v19, %[srdW], s40 offen\n\t" \
  "v_mfma_f32_16x16x32_f16 a[216:219], v[40:43], v[72:75], a[216:219]\n\t" \
  "v_cvt_pk_f16_f32 v122, v178, v182\n\t" \
  "v_mfma_f32_16x16x32_f16 a[212:215], v[40:43], v[68:71], a[212:215]\n\t" \
  "ds_read_b128 v[60:63], v25 offset:31744\n\t" \
  "v_mfma_f32_16x16x32_f16 a[212:215], v[44:47], v[84:87], a[212:215]\n\t" \
  "v_cvt_pk_f16_f32 v126, v179, v183\n\t" \
  "v_mfma_f32_16x16x32_f16 a[208:211], v[44:47], v[80:83], a[208:211]\n\t" \
  "buffer_load_dwordx4 v[176:179], v20, %[srdW], s40 offen\n\t" \
  "v_mfma_f32_16x16x32_f16 a[208:211], v[40:43], v[64:67], a[208:211]\n\t" \
  "s_waitcnt vmcnt(23)\n\t" \
  "v_cvt_pk_f16_f32 v115, v184, v188\n\t" \
  "s_waitcnt lgkmcnt(2)\n\t" \
  "v_mfma_f32_16x16x32_f16 a[224:227], v[48:51], v[64:67], a[224:227]\n\t" \
  "ds_read_b128 v[32:35], v25 offset:32768\n\t" \
  "v_mfma_f32_16x16x32_f16 a[224:227], v[52:55], v[80:83], a[224:227]\n\t" \
  "v_cvt_pk_f16_f32 v119, v185, v189\n\t" \
  "v_mfma_f32_16x16x32_f16 a[228:231], v[52:55], v[84:87], a[228:231]\n\t" \
  "buffer_load_dwordx4 v[180:183], v21, %[srdW], s40 offen\n\t" \
  "v_mfma_f32_16x16x32_f16 a[228:231], v[48:51], v[68:71], a[228:231]\n\t" \
  "v_cvt_pk_f16_f32 v123, v186, v190\n\t" \
  "v_mfma_f32_16x16x32_f16 a[232:235], v[48:51], v[72:75], a[232:235]\n\t" \
  "ds_read_b128 v[36:39], v25 offset:49152\n\t" \
  "v_mfma_f32_16x16x32_f16 a[232:235], v[52:55], v[88:91], a[232:235]\n\t" \
  "v_cvt_pk_f16_f32 v127, v187, v191\n\t" \
  "v_mfma_f32_16x16x32_f16 a[236:239], v[52:55], v[92:95], a[236:239]\n\t" \
  "buffer_load_dwordx4 v[184:187], v22, %[srdW], s40 offen\n\t" \
  "v_mfma_f32_16x16x32_f16 a[236:239], v[48:51], v[76:79], a[236:239]\n\t" \
  "v_cmp_le_u32 s[52:53], s59, v29\n\t" \
  "s_waitcnt lgkmcnt(2)\n\t" \
  "v_mfma_f32_16x16x32_f16 a[252:255], v[56:59], v[76:79], a[252:255]\n\t" \
  "ds_read_b128 v[40:43], v25 offset:33792\n\t" \
  "v_mfma_f32_16x16x32_f16 a[252:255], v[60:63], v[92:95], a[252:255]\n\t" \
  "s_and_b32 s47, s46, 0xffff\n\t" \
  "s_bitcmp1_b32 s47, 6\n\ts_cselect_b64 s[78:79], s[52:53], s[50:51]\n\t" \
  "v_mfma_f32_16x16x32_f16 a[248:251], v[60:63], v[88:91], a[248:251]\n\t" \
  "buffer_load_dwordx4 v[188:191], v23, %[srdW], s40 offen\n\t" \
  "v_mfma_f32_16x16x32_f16 a[248:251], v[56:59], v[72:75], a[248:251]\n\t" \
  "s_lshr_b32 s40, s47, 7\n\t" \
  "s_lshl_b32 s42, s40, 16\n\t" \
  "v_mfma_f32_16x16x32_f16 a[244:247], v[56:59], v[68:71], a[244:247]\n\t" \
  "ds_read_b128 v[44:47], v25 offset:50176\n\t" \
  "s_lshl_b32 s40, s40, 20\n\t" \
  "v_mfma_f32_16x16x32_f16 a[244:247], v[60:63], v[84:87], a[244:247]\n\t" \
  "s_add_u32 s43, s43, 0x4000\n\t" \
  "s_and_b32 m0, s43, 0x1ffff\n\t" \
  "v_mfma_f32_16x16x32_f16 a[240:243], v[60:63], v[80:83], a[240:243]\n\t" \
  "buffer_load_dwordx2 v[28:29], %[vFo], %[srdC], 0 offen sc0 sc1\n\t" \
  "v_mfma_f32_16x16x32_f16 a[240:243], v[56:59], v[64:67], a[240:243]\n\t" \
  "s_bitcmp1_b64 s[78:79], s47\n\ts_cbranch_scc0 LSLOWC1_%=\nLBACKC1_%=:\n\t" \
  "s_branch LN2_%=\n\t" \
  ".p2align 6\n\t" \
  "LC2_%=:\n\t" \
  "s_waitcnt lgkmcnt(2)\n\t" \
  "v_mfma_f32_16x16x32_f16 a[0:3], v[32:35], v[96:99], a[0:3]\n\t" \
  "ds_read_b128 v[48:51], v25 offset:34816\n\t" \
  "v_readlane_b32 s54, %[vDlo], s49\n\t" \
  "v_mfma_f32_16x16x32_f16 a[0:3], v[36:39], v[112:115], a[0:3]\n\t" \
  "v_readlane_b32 s55, %[vDhi], s49\n\t" \
  "v_readlane_b32 s56, %[vGlo], s49\n\t" \
  "v_mfma_f32_16x16x32_f16 a[4:7], v[36:39], v[116:119], a[4:7]\n\t" \
  "buffer_load_dwordx4 v24, %[srdA], s42 offen offset:0 sc0 sc1 lds\n\t" \
  "v_mfma_f32_16x16x32_f16 a[4:7], v[32:35], v[100:103], a[4:7]\n\t" \
  "v_readlane_b32 s57, %[vGhi], s49\n\t" \
  "v_readlane_b32 s58, %[vXo], s49\n\t" \
  "v_mfma_f32_16x16x32_f16 a[8:11], v[32:35], v[104:107], a[8:11]\n\t" \
  "ds_read_b128 v[52:55], v25 offset:51200\n\t" \
  "v_mfma_f32_16x16x32_f16 a[8:11], v[36:39], v[120:123], a[8:11]\n\t" \
  "s_add_u32 s49, s49, 1\n\t" \
  "s_cmp_lg_u32 s55, 0\n\ts_cselect_b64 s[64:65], -1, 0\n\t" \
  "v_mfma_f32_16x16x32_f16 a[12:15], v[36:39], v[124:127], a[12:15]\n\t" \
  "buffer_load_dwordx4 v24, %[srdA], s42 offen offset:1024 sc0 sc1 lds\n\t" \
  "v_mfma_f32_16x16x32_f16 a[12:15], v[32:35], v[108:111], a[12:15]\n\t" \
  "s_waitcnt lgkmcnt(2)\n\t" \
  "v_mfma_f32_16x16x32_f16 a[28:31], v[40:43], v[108:111], a[28:31]\n\t" \
  "ds_read_b128 v[56:59], v25 offset:35840\n\t" \
  "v_mfma_f32_16x16x32_f16 a[28:31], v[44:47], v[124:127], a[28:31]\n\t" \
  "s_cmp_lg_u32 s57, 0\n\ts_cselect_b64 s[66:67], 1, 0\n\t" \
  "s_cmp_ge_u32 s49, %[sZN]\n\ts_cselect_b32 s48, 99, s48\n\t" \
  "v_mfma_f32_16x16x32_f16 a[24:27], v[44:47], v[120:123], a[24:27]\n\t" \
  "buffer_load_dwordx4 v24, %[srdA], s42 offen offset:2048 sc0 sc1 lds\n\t" \
  "v_mfma_f32_16x16x32_f16 a[24:27], v[40:43], v[104:107], a[24:27]\n\t" \
  "v_cvt_pk_f16_f32 v200, v192, v193\n\t" \
  "v_mfma_f32_16x16x32_f16 a[20:23], v[40:43], v[100:103], a[20:23]\n\t" \
  "ds_read_b128 v[60:63], v25 offset:52224\n\t" \
  "v_mfma_f32_16x16x32_f16 a[20:23], v[44:47], v[116:119], a[20:23]\n\t" \
  "v_cvt_pk_f16_f32 v201, v194, v195\n\t" \
  "v_cvt_pk_f16_f32 v202, v196, v197\n\t" \
  "v_mfma_f32_16x16x32_f16 a[16:19], v[44:47], v[112:115], a[16:19]\n\t" \
  "buffer_load_dwordx4 v24, %[srdA], s42 offen offset:3072 sc0 sc1 lds\n\t" \
  "v_mfma_f32_16x16x32_f16 a[16:19], v[40:43], v[96:99], a[16:19]\n\t" \
  "v_cvt_pk_f16_f32 v203, v198, v199\n\t" \
  "s_waitcnt lgkmcnt(2)\n\t" \
  "v_mfma_f32_16x16x32_f16 a[32:35], v[48:51], v[96:99], a[32:35]\n\t" \
  "ds_read_b128 v[32:35], v25 offset:36864\n\t" \
  "v_mfma_f32_16x16x32_f16 a[32:35], v[52:55], v[112:115], a[32:35]\n\t" \
  "s_mov_b64 exec, s[66:67]\n\tglobal_store_dword v31, v27, s[56:57] sc0 sc1\n\ts_mov_b64 exec, -1\n\t" \
  "v_mfma_f32_16x16x32_f16 a[36:39], v[52:55], v[116:119], a[36:39]\n\t" \
  "s_mov_b64 exec, s[64:65]\n\tglobal_store_dwordx4 v26, v[200:203], s[54:55] sc0 sc1\n\ts_mov_b64 exec, -1\n\t" \
  "v_mfma_f32_16x16x32_f16 a[36:39], v[48:51], v[100:103], a[36:39]\n\t" \
  "v_mfma_f32_16x16x32_f16 a[40:43], v[48:51], v[104:107], a[40:43]\n\t" \
  "ds_read_b128 v[36:39], v25 offset:53248\n\t" \
  "v_mfma_f32_16x16x32_f16 a[40:43], v[52:55], v[120:123], a[40:43]\n\t" \
  "v_cmp_le_u32 s[50:51], s59, v28\n\t" \
  "v_mfma_f32_16x16x32_f16 a[44:47], v[52:55], v[124:127], a[44:47]\n\t" \
  "buffer_load_dwordx4 v[192:195], %[vXa], %[srdX], s58 offen\n\t" \
  "v_mfma_f32_16x16x32_f16 a[44:47], v[48:51], v[108:111], a[44:47]\n\t" \
  "s_waitcnt vmcnt(23)\n\t" \
  "v_cvt_pk_f16_f32 v64, v128, v132\n\t" \
  "s_waitcnt lgkmcnt(2)\n\t" \
  "v_mfma_f32_16x16x32_f16 a[60:63], v[56:59], v[108:111], a[60:63]\n\t" \
  "ds_read_b128 v[40:43], v25 offset:37888\n\t" \
  "v_mfma_f32_16x16x32_f16 a[60:63], v[60:63], v[124:127], a[60:63]\n\t" \
  "v_cvt_pk_f16_f32 v68, v129, v133\n\t" \
  "v_mfma_f32_16x16x32_f16 a[56:59], v[60:63], v[120:123], a[56:59]\n\t" \
  "buffer_load_dwordx4 v[196:199], %[vXa], %[srdX], s58 offen offset:64\n\t" \
  "v_mfma_f32_16x16x32_f16 a[56:59], v[56:59], v[104:107], a[56:59]\n\t" \
  "v_cvt_pk_f16_f32 v72, v130, v134\n\t" \
  "v_mfma_f32_16x16x32_f16 a[52:55], v[56:59], v[100:103], a[52:55]\n\t" \
  "ds_read_b128 v[44:47], v25 offset:54272\n\t" \
  "v_mfma_f32_16x16x32_f16 a[52:55], v[60:63], v[116:119], a[52:55]\n\t" \
  "v_cvt_pk_f16_f32 v76, v131, v135\n\t" \
  "v_mfma_f32_16x16x32_f16 a[48:51], v[60:63], v[112:115], a[48:51]\n\t" \
  "buffer_load_dwordx4 v[128:131], v16, %[srdW], s40 offen\n\t" \
  "v_mfma_f32_16x16x32_f16 a[48:51], v[56:59], v[96:99], a[48:51]\n\t" \
  "s_waitcnt vmcnt(23)\n\t" \
  "v_cvt_pk_f16_f32 v65, v136, v140\n\t" \
  "s_waitcnt lgkmcnt(2)\n\t" \
  "v_mfma_f32_16x16x32_f16 a[64:67], v[32:35], v[96:99], a[64:67]\n\t" \
  "ds_read_b128 v[48:51], v25 offset:38912\n\t" \
  "v_mfma_f32_16x16x32_f16 a[64:67], v[36:39], v[112:115], a[64:67]\n\t" \
  "v_cvt_pk_f16_f32 v69, v137, v141\n\t" \
  "v_mfma_f32_16x16x32_f16 a[68:71], v[36:39], v[116:119], a[68:71]\n\t" \
  "buffer_load_dwordx4 v[132:135], v17, %[srdW], s40 offen\n\t" \
  "v_mfma_f32_16x16x32_f16 a[68:71], v[32:35], v[100:103], a[68:71]\n\t" \
  "v_cvt_pk_f16_f32 v73, v138, v142\n\t" \
  "v_mfma_f32_16x16x32_f16 a[72:75], v[32:35], v[104:107], a[72:75]\n\t" \
  "ds_read_b128 v[52:55], v25 offset:55296\n\t" \
  "v_mfma_f32_16x16x32_f16 a[72:75], v[36:39], v[120:123], a[72:75]\n\t" \
  "v_cvt_pk_f16_f32 v77, v139, v143\n\t" \
  "v_mfma_f32_16x16x32_f16 a[76:79], v[36:39], v[124:127], a[76:79]\n\t" \
  "buffer_load_dwordx4 v[136:139], v18, %[srdW], s40 offen\n\t" \
  "v_mfma_f32_16x16x32_f16 a[76:79], v[32:35], v[108:111], a[76:79]\n\t" \
  "s_waitcnt vmcnt(23)\n\t" \
  "v_cvt_pk_f16_f32 v66, v144, v148\n\t" \
  "s_waitcnt lgkmcnt(2)\n\t" \
  "v_mfma_f32_16x16x32_f16 a[92:95], v[40:43], v[108:111], a[92:95]\n\t" \
  "ds_read_b128 v[56:59], v25 offset:39936\n\t" \
  "v_mfma_f32_16x16x32_f16 a[92:95], v[44:47], v[124:127], a[92:95]\n\t" \
  "v_cvt_pk_f16_f32 v70, v145, v149\n\t" \
  "v_mfma_f32_16x16x32_f16 a[88:91], v[44:47], v[120:123], a[88:91]\n\t" \
  "buffer_load_dwordx4 v[140:143], v19, %[srdW], s40 offen\n\t" \
  "v_mfma_f32_16x16x32_f16 a[88:91], v[40:43], v[104:107], a[88:91]\n\t" \
  "v_cvt_pk_f16_f32 v74, v146, v150\n\t" \
  "v_mfma_f32_16x16x32_f16 a[84:87], v[40:43], v[100:103], a[84:87]\n\t" \
  "ds_read_b128 v[60:63], v25 offset:56320\n\t" \
  "v_mfma_f32_16x16x32_f16 a[84:87], v[44:47], v[116:119], a[84:87]\n\t" \
  "v_cvt_pk_f16_f32 v78, v147, v151\n\t" \
  "v_mfma_f32_16x16x32_f16 a[80:83], v[44:47], v[112:115], a[80:83]\n\t" \
  "buffer_load_dwordx4 v[144:147], v20, %[srdW], s40 offen\n\t" \
  "v_mfma_f32_16x16x32_f16 a[80:83], v[40:43], v[96:99], a[80:83]\n\t" \
  "s_waitcnt vmcnt(23)\n\t" \
  "v_cvt_pk_f16_f32 v67, v152, v156\n\t" \
  "s_waitcnt lgkmcnt(2)\n\t" \
  "v_mfma_f32_16x16x32_f16 a[96:99], v[48:51], v[96:99], a[96:99]\n\t" \
  "ds_read_b128 v[32:35], v25 offset:40960\n\t" \
  "v_mfma_f32_16x16x32_f16 a[96:99], v[52:55], v[112:115], a[96:99]\n\t" \
  "v_cvt_pk_f16_f32 v71, v153, v157\n\t" \
  "v_mfma_f32_16x16x32_f16 a[100:103], v[52:55], v[116:119], a[100:103]\n\t" \
  "buffer_load_dwordx4 v[148:151], v21, %[srdW], s40 offen\n\t" \
  "v_mfma_f32_16x16x32_f16 a[100:103], v[48:51], v[100:103], a[100:103]\n\t" \
  "v_cvt_pk_f16_f32 v75, v154, v158\n\t" \
  "v_mfma_f32_16x16x32_f16 a[104:107], v[48:51], v[104:107], a[104:107]\n\t" \
  "ds_read_b128 v[36:39], v25 offset:57344\n\t" \
  "v_mfma_f32_16x16x32_f16 a[104:107], v[52:55], v[120:123], a[104:107]\n\t" \
  "v_cvt_pk_f16_f32 v79, v155, v159\n\t" \
  "v_mfma_f32_16x16x32_f16 a[108:111], v[52:55], v[124:127], a[108:111]\n\t" \
  "buffer_load_dwordx4 v[152:155], v22, %[srdW], s40 offen\n\t" \
  "v_mfma_f32_16x16x32_f16 a[108:111], v[48:51], v[108:111], a[108:111]\n\t" \
  "v_cmp_le_u32 s[52:53], s59, v29\n\t" \
  "s_waitcnt lgkmcnt(2)\n\t" \
  "v_mfma_f32_16x16x32_f16 a[124:127], v[56:59], v[108:111], a[124:127]\n\t" \
  "ds_read_b128 v[40:43], v25 offset:41984\n\t" \
  "v_mfma_f32_16x16x32_f16 a[124:127], v[60:63], v[124:127], a[124:127]\n\t" \
  "s_lshr_b32 s47, s46, 16\n\t" \
  "s_bitcmp1_b32 s47, 6\n\ts_cselect_b64 s[78:79], s[52:53], s[50:51]\n\t" \
  "v_mfma_f32_16x16x32_f16 a[120:123], v[60:63], v[120:123], a[120:123]\n\t" \
  "buffer_load_dwordx4 v[156:159], v23, %[srdW], s40 offen\n\t" \
  "v_mfma_f32_16x16x32_f16 a[120:123], v[56:59], v[104:107], a[120:123]\n\t" \
  "s_lshr_b32 s40, s47, 7\n\t" \
  "s_lshl_b32 s42, s40, 16\n\t" \
  "v_mfma_f32_16x16x32_f16 a[116:119], v[56:59], v[100:103], a[116:119]\n\t" \
  "ds_read_b128 v[44:47], v25 offset:58368\n\t" \
  "s_lshl_b32 s40, s40, 20\n\t" \
  "v_mfma_f32_16x16x32_f16 a[116:119], v[60:63], v[116:119], a[116:119]\n\t" \
  "s_add_u32 s43, s43, 0x4000\n\t" \
  "s_and_b32 m0, s43, 0x1ffff\n\t" \
  "v_mfma_f32_16x16x32_f16 a[112:115], v[60:63], v[112:115], a[112:115]\n\t" \
  "buffer_load_dwordx2 v[28:29], %[vFo], %[srdC], 0 offen sc0 sc1\n\t" \
  "v_mfma_f32_16x16x32_f16 a[112:115], v[56:59], v[96:99], a[112:115]\n\t" \
  "s_bitcmp1_b64 s[78:79], s47\n\ts_cbranch_scc0 LSLOWC2_%=\nLBACKC2_%=:\n\t" \
  "s_branch LN3_%=\n\t" \
  ".p2align 6\n\t" \
  "LC3_%=:\n\t" \
  "s_waitcnt lgkmcnt(2)\n\t" \
  "v_mfma_f32_16x16x32_f16 a[128:131], v[32:35], v[96:99], a[128:131]\n\t" \
  "ds_read_b128 v[48:51], v25 offset:43008\n\t" \
  "v_readlane_b32 s54, %[vDlo], s49\n\t" \
  "v_mfma_f32_16x16x32_f16 a[128:131], v[36:39], v[112:115], a[128:131]\n\t" \
  "v_readlane_b32 s55, %[vDhi], s49\n\t" \
  "v_readlane_b32 s56, %[vGlo], s49\n\t" \
  "v_mfma_f32_16x16x32_f16 a[132:135], v[36:39], v[116:119], a[132:135]\n\t" \
  "buffer_load_dwordx4 v24, %[srdA], s42 offen offset:0 sc0 sc1 lds\n\t" \
  "v_mfma_f32_16x16x32_f16 a[132:135], v[32:35], v[100:103], a[132:135]\n\t" \
  "v_readlane_b32 s57, %[vGhi], s49\n\t" \
  "v_readlane_b32 s58, %[vXo], s49\n\t" \
  "v_mfma_f32_16x16x32_f16 a[136:139], v[32:35], v[104:107], a[136:139]\n\t" \
  "ds_read_b128 v[52:55], v25 offset:59392\n\t" \
  "v_mfma_f32_16x16x32_f16 a[136:139], v[36:39], v[120:123], a[136:139]\n\t" \
  "s_add_u32 s49, s49, 1\n\t" \
  "s_cmp_lg_u32 s55, 0\n\ts_cselect_b64 s[64:65], -1, 0\n\t" \
  "v_mfma_f32_16x16x32_f16 a[140:143], v[36:39], v[124:127], a[140:143]\n\t" \
  "buffer_load_dwordx4 v24, %[srdA], s42 offen offset:1024 sc0 sc1 lds\n\t" \
  "v_mfma_f32_16x16x32_f16 a[140:143], v[32:35], v[108:111], a[140:143]\n\t" \
  "s_waitcnt lgkmcnt(2)\n\t" \
  "v_mfma_f32_16x16x32_f16 a[156:159], v[40:43], v[108:111], a[156:159]\n\t" \
  "ds_read_b128 v[56:59], v25 offset:44032\n\t" \
  "v_mfma_f32_16x16x32_f16 a[156:159], v[44:47], v[124:127], a[156:159]\n\t" \
  "s_cmp_lg_u32 s57, 0\n\ts_cselect_b64 s[66:67], 1, 0\n\t" \
  "s_cmp_ge_u32 s49, %[sZN]\n\ts_cselect_b32 s48, 99, s48\n\t" \
  "v_mfma_f32_16x16x32_f16 a[152:155], v[44:47], v[120:123], a[152:155]\n\t" \
  "buffer_load_dwordx4 v24, %[srdA], s42 offen offset:2048 sc0 sc1 lds\n\t" \
  "v_mfma_f32_16x16x32_f16 a[152:155], v[40:43], v[104:107], a[152:155]\n\t" \
  "s_waitcnt vmcnt(21)\n\t" \
  "s_barrier\n\t" \
  "v_cvt_pk_f16_f32 v200, v192, v193\n\t" \
  "v_mfma_f32_16x16x32_f16 a[148:151], v[40:43], v[100:103], a[148:151]\n\t" \
  "ds_read_b128 v[60:63], v25 offset:60416\n\t" \
  "v_mfma_f32_16x16x32_f16 a[148:151], v[44:47], v[116:119], a[148:151]\n\t" \
  "v_cvt_pk_f16_f32 v201, v194, v195\n\t" \
  "v_cvt_pk_f16_f32 v202, v196, v197\n\t" \
  "v_mfma_f32_16x16x32_f16 a[144:147], v[44:47], v[112:115], a[144:147]\n\t" \
  "buffer_load_dwordx4 v24, %[srdA], s42 offen offset:3072 sc0 sc1 lds\n\t" \
  "v_mfma_f32_16x16x32_f16 a[144:147], v[40:43], v[96:99], a[144:147]\n\t" \
  "v_cvt_pk_f16_f32 v203, v198, v199\n\t" \
  "s_waitcnt lgkmcnt(2)\n\t" \
  "v_mfma_f32_16x16x32_f16 a[160:163], v[48:51], v[96:99], a[160:163]\n\t" \
  "ds_read_b128 v[32:35], v25 offset:45056\n\t" \
  "v_mfma_f32_16x16x32_f16 a[160:163], v[52:55], v[112:115], a[160:163]\n\t" \
  "s_mov_b64 exec, s[66:67]\n\tglobal_store_dword v31, v27, s[56:57] sc0 sc1\n\ts_mov_b64 exec, -1\n\t" \
  "v_mfma_f32_16x16x32_f16 a[164:167], v[52:55], v[116:119], a[164:167]\n\t" \
  "s_mov_b64 exec, s[64:65]\n\tglobal_store_dwordx4 v26, v[200:203], s[54:55] sc0 sc1\n\ts_mov_b64 exec, -1\n\t" \
  "v_mfma_f32_16x16x32_f16 a[164:167], v[48:51], v[100:103], a[164:167]\n\t" \
  "v_readlane_b32 s46, %[vTab], s44\n\t" \
  "v_mfma_f32_16x16x32_f16 a[168:171], v[48:51], v[104:107], a[168:171]\n\t" \
  "ds_read_b128 v[36:39], v25 offset:61440\n\t" \
  "v_mfma_f32_16x16x32_f16 a[168:171], v[52:55], v[120:123], a[168:171]\n\t" \
  "s_add_u32 s44, s44, 1\n\t" \
  "v_cmp_le_u32 s[50:51], s59, v28\n\t" \
  "v_mfma_f32_16x16x32_f16 a[172:175], v[52:55], v[124:127], a[172:175]\n\t" \
  "buffer_load_dwordx4 v[192:195], %[vXa], %[srdX], s58 offen\n\t" \
  "v_mfma_f32_16x16x32_f16 a[172:175], v[48:51], v[108:111], a[172:175]\n\t" \
  "v_cvt_pk_f16_f32 v80, v160, v164\n\t" \
  "s_waitcnt lgkmcnt(2)\n\t" \
  "v_mfma_f32_16x16x32_f16 a[188:191], v[56:59], v[108:111], a[188:191]\n\t" \
  "ds_read_b128 v[40:43], v25 offset:46080\n\t" \
  "v_mfma_f32_16x16x32_f16 a[188:191], v[60:63], v[124:127], a[188:191]\n\t" \
  "v_cvt_pk_f16_f32 v84, v161, v165\n\t" \
  "v_mfma_f32_16x16x32_f16 a[184:187], v[60:63], v[120:123], a[184:187]\n\t" \
  "buffer_load_dwordx4 v[196:199], %[vXa], %[srdX], s58 offen offset:64\n\t" \
  "v_mfma_f32_16x16x32_f16 a[184:187], v[56:59], v[104:107], a[184:187]\n\t" \
  "v_cvt_pk_f16_f32 v88, v162, v166\n\t" \
  "v_mfma_f32_16x16x32_f16 a[180:183], v[56:59], v[100:103], a[180:183]\n\t" \
  "ds_read_b128 v[44:47], v25 offset:62464\n\t" \
  "v_mfma_f32_16x16x32_f16 a[180:183], v[60:63], v[116:119], a[180:183]\n\t" \
  "v_cvt_pk_f16_f32 v92, v163, v167\n\t" \
  "v_mfma_f32_16x16x32_f16 a[176:179], v[60:63], v[112:115], a[176:179]\n\t" \
  "buffer_load_dwordx4 v[160:163], v16, %[srdW], s40 offen\n\t" \
  "v_mfma_f32_16x16x32_f16 a[176:179], v[56:59], v[96:99], a[176:179]\n\t" \
  "s_waitcnt vmcnt(23)\n\t" \
  "v_cvt_pk_f16_f32 v81, v168, v172\n\t" \
  "s_waitcnt lgkmcnt(2)\n\t" \
  "v_mfma_f32_16x16x32_f16 a[192:195], v[32:35], v[96:99], a[192:195]\n\t" \
  "ds_read_b128 v[48:51], v25 offset:47104\n\t" \
  "v_mfma_f32_16x16x32_f16 a[192:195], v[36:39], v[112:115], a[192:195]\n\t" \
  "v_cvt_pk_f16_f32 v85, v169, v173\n\t" \
  "v_mfma_f32_16x16x32_f16 a[196:199], v[36:39], v[116:119], a[196:199]\n\t" \
  "buffer_load_dwordx4 v[164:167], v17, %[srdW], s40 offen\n\t" \
  "v_mfma_f32_16x16x32_f16 a[196:199], v[32:35], v[100:103], a[196:199]\n\t" \
  "v_cvt_pk_f16_f32 v89, v170, v174\n\t" \
  "v_mfma_f32_16x16x32_f16 a[200:203], v[32:35], v[104:107], a[200:203]\n\t" \
  "ds_read_b128 v[52:55], v25 offset:63488\n\t" \
  "v_mfma_f32_16x16x32_f16 a[200:203], v[36:39], v[120:123], a[200:203]\n\t" \
  "v_cvt_pk_f16_f32 v93, v171, v175\n\t" \
  "v_mfma_f32_16x16x32_f16 a[204:207], v[36:39], v[124:127], a[204:207]\n\t" \
  "buffer_load_dwordx4 v[168:171], v18, %[srdW], s40 offen\n\t" \
  "v_mfma_f32_16x16x32_f16 a[204:207], v[32:35], v[108:111], a[204:207]\n\t" \
  "s_waitcnt vmcnt(23)\n\t" \
  "v_cvt_pk_f16_f32 v82, v176, v180\n\t" \
  "s_waitcnt lgkmcnt(2)\n\t" \
  "v_mfma_f32_16x16x32_f16 a[220:223], v[40:43], v[108:111], a[220:223]\n\t" \
  "ds_read_b128 v[56:59], v25 offset:48128\n\t" \
  "v_mfma_f32_16x16x32_f16 a[220:223], v[44:47], v[124:127], a[220:223]\n\t" \
  "v_cvt_pk_f16_f32 v86, v177, v181\n\t" \
  "v_mfma_f32_16x16x32_f16 a[216:219], v[44:47], v[120:123], a[216:219]\n\t" \
  "buffer_load_dwordx4 v[172:175], v19, %[srdW], s40 offen\n\t" \
  "v_mfma_f32_16x16x32_f16 a[216:219], v[40:43], v[104:107], a[216:219]\n\t" \
  "v_cvt_pk_f16_f32 v90, v178, v182\n\t" \
  "v_mfma_f32_16x16x32_f16 a[212:215], v[40:43], v[100:103], a[212:215]\n\t" \
  "ds_read_b128 v[60:63], v25 offset:64512\n\t" \
  "v_mfma_f32_16x16x32_f16 a[212:215], v[44:47], v[116:119], a[212:215]\n\t" \
  "v_cvt_pk_f16_f32 v94, v179, v183\n\t" \
  "v_mfma_f32_16x16x32_f16 a[208:211], v[44:47], v[112:115], a[208:211]\n\t" \
  "buffer_load_dwordx4 v[176:179], v20, %[srdW], s40 offen\n\t" \
  "v_mfma_f32_16x16x32_f16 a[208:211], v[40:43], v[96:99], a[208:211]\n\t" \
  "v_xor_b32 v25, 0x10000, v25\n\t" \
  "s_waitcnt vmcnt(23)\n\t" \
  "v_cvt_pk_f16_f32 v83, v184, v188\n\t" \
  "s_waitcnt lgkmcnt(2)\n\t" \
  "v_mfma_f32_16x16x32_f16 a[224:227], v[48:51], v[96:99], a[224:227]\n\t" \
  "ds_read_b128 v[32:35], v25 offset:0\n\t" \
  "v_mfma_f32_16x16x32_f16 a[224:227], v[52:55], v[112:115], a[224:227]\n\t" \
  "v_cvt_pk_f16_f32 v87, v185, v189\n\t" \
  "v_mfma_f32_16x16x32_f16 a[228:231], v[52:55], v[116:119], a[228:231]\n\t" \
  "buffer_load_dwordx4 v[180:183], v21, %[srdW], s40 offen\n\t" \
  "v_mfma_f32_16x16x32_f16 a[228:231], v[48:51], v[100:103], a[228:231]\n\t" \
  "v_cvt_pk_f16_f32 v91, v186, v190\n\t" \
  "v_mfma_f32_16x16x32_f16 a[232:235], v[48:51], v[104:107], a[232:235]\n\t" \
  "ds_read_b128 v[36:39], v25 offset:16384\n\t" \
  "v_mfma_f32_16x16x32_f16 a[232:235], v[52:55], v[120:123], a[232:235]\n\t" \
  "v_cvt_pk_f16_f32 v95, v187, v191\n\t" \
  "v_mfma_f32_16x16x32_f16 a[236:239], v[52:55], v[124:127], a[236:239]\n\t" \
  "buffer_load_dwordx4 v[184:187], v22, %[srdW], s40 offen\n\t" \
  "v_mfma_f32_16x16x32_f16 a[236:239], v[48:51], v[108:111], a[236:239]\n\t" \
  "v_cmp_le_u32 s[52:53], s59, v29\n\t" \
  "s_waitcnt lgkmcnt(2)\n\t" \
  "v_mfma_f32_16x16x32_f16 a[252:255], v[56:59], v[108:111], a[252:255]\n\t" \
  "ds_read_b128 v[40:43], v25 offset:1024\n\t" \
  "v_mfma_f32_16x16x32_f16 a[252:255], v[60:63], v[124:127], a[252:255]\n\t" \
  "s_and_b32 s47, s46, 0xffff\n\t" \
  "s_bitcmp1_b32 s47, 6\n\ts_cselect_b64 s[78:79], s[52:53], s[50:51]\n\t" \
  "v_mfma_f32_16x16x32_f16 a[248:251], v[60:63], v[120:123], a[248:251]\n\t" \
  "buffer_load_dwordx4 v[188:191], v23, %[srdW], s40 offen\n\t" \
  "v_mfma_f32_16x16x32_f16 a[248:251], v[56:59], v[104:107], a[248:251]\n\t" \
  "s_lshr_b32 s40, s47, 7\n\t" \
  "s_lshl_b32 s42, s40, 16\n\t" \
  "v_mfma_f32_16x16x32_f16 a[244:247], v[56:59], v[100:103], a[244:247]\n\t" \
  "ds_read_b128 v[44:47], v25 offset:17408\n\t" \
  "s_lshl_b32 s40, s40, 20\n\t" \
  "v_mfma_f32_16x16x32_f16 a[244:247], v[60:63], v[116:119], a[244:247]\n\t" \
  "s_add_u32 s43, s43, 0x4000\n\t" \
  "s_and_b32 m0, s43, 0x1ffff\n\t" \
  "v_mfma_f32_16x16x32_f16 a[240:243], v[60:63], v[112:115], a[240:243]\n\t" \
  "buffer_load_dwordx2 v[28:29], %[vFo], %[srdC], 0 offen sc0 sc1\n\t" \
  "v_mfma_f32_16x16x32_f16 a[240:243], v[56:59], v[96:99], a[240:243]\n\t" \
  "s_bitcmp1_b64 s[78:79], s47\n\ts_cbranch_scc0 LSLOWC3_%=\nLBACKC3_%=:\n\t" \
  "s_branch LN4_%=\n\t" \
  "LSLOWN0_%=:\n\t" \
  "s_memrealtime s[60:61]\n\t" \
  "s_add_u32 s69, s69, 1\n\t" \
  "LSPINN0_%=:\n\t" \
  "buffer_load_dwordx2 v[28:29], %[vFo], %[srdC], 0 offen sc0 sc1\n\t" \
  "s_waitcnt vmcnt(0) lgkmcnt(0)\n\t" \
  "v_cmp_le_u32 s[50:51], s59, v28\n\t" \
  "v_cmp_le_u32 s[52:53], s59, v29\n\t" \
  "s_nop 1\n\t" \
  "s_bitcmp1_b32 s47, 6\n\ts_cselect_b64 s[78:79], s[52:53], s[50:51]\n\t" \
  "s_bitcmp1_b64 s[78:79], s47\n\t" \
  "s_cbranch_scc1 LBACKN0_%=\n\t" \
  "s_sleep 2\n\t" \
  "s_memrealtime s[62:63]\n\t" \
  "s_waitcnt lgkmcnt(0)\n\t" \
  "s_sub_u32 s62, s62, s60\n\t" \
  "s_cmp_lt_u32 s62, %[sDL]\n\t" \
  "s_cbranch_scc1 LSPINN0_%=\n\t" \
  "v_mov_b32 v30, %[cTMO]\n\t" \
  "buffer_store_dword v27, v30, %[srdC], 0 offen sc1\n\t" \
  "s_waitcnt vmcnt(0)\n\t" \
  "s_mov_b32 s59, 0\n\t" \
  "s_branch LBACKN0_%=\n\t" \
  "LSLOWN1_%=:\n\t" \
  "s_memrealtime s[60:61]\n\t" \
  "s_add_u32 s69, s69, 1\n\t" \
  "LSPINN1_%=:\n\t" \
  "buffer_load_dwordx2 v[28:29], %[vFo], %[srdC], 0 offen sc0 sc1\n\t" \
  "s_waitcnt vmcnt(0) lgkmcnt(0)\n\t" \
  "v_cmp_le_u32 s[50:51], s59, v28\n\t" \
  "v_cmp_le_u32 s[52:53], s59, v29\n\t" \
  "s_nop 1\n\t" \
  "s_bitcmp1_b32 s47, 6\n\ts_cselect_b64 s[78:79], s[52:53], s[50:51]\n\t" \
  "s_bitcmp1_b64 s[78:79], s47\n\t" \
  "s_cbranch_scc1 LBACKN1_%=\n\t" \
  "s_sleep 2\n\t" \
  "s_memrealtime s[62:63]\n\t" \
  "s_waitcnt lgkmcnt(0)\n\t" \
  "s_sub_u32 s62, s62, s60\n\t" \
  "s_cmp_lt_u32 s62, %[sDL]\n\t" \
  "s_cbranch_scc1 LSPINN1_%=\n\t" \
  "v_mov_b32 v30, %[cTMO]\n\t" \
  "buffer_store_dword v27, v30, %[srdC], 0 offen sc1\n\t" \
  "s_waitcnt vmcnt(0)\n\t" \
  "s_mov_b32 s59, 0\n\t" \
  "s_branch LBACKN1_%=\n\t" \
  "LSLOWN2_%=:\n\t" \
  "s_memrealtime s[60:61]\n\t" \
  "s_add_u32 s69, s69, 1\n\t" \
  "LSPINN2_%=:\n\t" \
  "buffer_load_dwordx2 v[28:29], %[vFo], %[srdC], 0 offen sc0 sc1\n\t" \
  "s_waitcnt vmcnt(0) lgkmcnt(0)\n\t" \
  "v_cmp_le_u32 s[50:51], s59, v28\n\t" \
  "v_cmp_le_u32 s[52:53], s59, v29\n\t" \
  "s_nop 1\n\t" \
  "s_bitcmp1_b32 s47, 6\n\ts_cselect_b64 s[78:79], s[52:53], s[50:51]\n\t" \
  "s_bitcmp1_b64 s[78:79], s47\n\t" \
  "s_cbranch_scc1 LBACKN2_%=\n\t" \
  "s_sleep 2\n\t" \
  "s_memrealtime s[62:63]\n\t" \
  "s_waitcnt lgkmcnt(0)\n\t" \
  "s_sub_u32 s62, s62, s60\n\t" \
  "s_cmp_lt_u32 s62, %[sDL]\n\t" \
  "s_cbranch_scc1 LSPINN2_%=\n\t" \
  "v_mov_b32 v30, %[cTMO]\n\t" \
  "buffer_store_dword v27, v30, %[srdC], 0 offen sc1\n\t" \
  "s_waitcnt vmcnt(0)\n\t" \
  "s_mov_b32 s59, 0\n\t" \
  "s_branch LBACKN2_%=\n\t" \
  "LSLOWN3_%=:\n\t" \
  "s_memrealtime s[60:61]\n\t" \
  "s_add_u32 s69, s69, 1\n\t" \
  "LSPINN3_%=:\n\t" \
  "buffer_load_dwordx2 v[28:29], %[vFo], %[srdC], 0 offen sc0 sc1\n\t" \
  "s_waitcnt vmcnt(0) lgkmcnt(0)\n\t" \
  "v_cmp_le_u32 s[50:51], s59, v28\n\t" \
  "v_cmp_le_u32 s[52:53], s59, v29\n\t" \
  "s_nop 1\n\t" \
  "s_bitcmp1_b32 s47, 6\n\ts_cselect_b64 s[78:79], s[52:53], s[50:51]\n\t" \
  "s_bitcmp1_b64 s[78:79], s47\n\t" \
  "s_cbranch_scc1 LBACKN3_%=\n\t" \
  "s_sleep 2\n\t" \
  "s_memrealtime s[62:63]\n\t" \
  "s_waitcnt lgkmcnt(0)\n\t" \
  "s_sub_u32 s62, s62, s60\n\t" \
  "s_cmp_lt_u32 s62, %[sDL]\n\t" \
  "s_cbranch_scc1 LSPINN3_%=\n\t" \
  "v_mov_b32 v30, %[cTMO]\n\t" \
  "buffer_store_dword v27, v30, %[srdC], 0 offen sc1\n\t" \
  "s_waitcnt vmcnt(0)\n\t" \
  "s_mov_b32 s59, 0\n\t" \
  "s_branch LBACKN3_%=\n\t" \
  "LSLOWC0_%=:\n\t" \
  "s_memrealtime s[60:61]\n\t" \
  "s_add_u32 s69, s69, 1\n\t" \
  "LSPINC0_%=:\n\t" \
  "buffer_load_dwordx2 v[28:29], %[vFo], %[srdC], 0 offen sc0 sc1\n\t" \
  "s_waitcnt vmcnt(0) lgkmcnt(0)\n\t" \
  "v_cmp_le_u32 s[50:51], s59, v28\n\t" \
  "v_cmp_le_u32 s[52:53], s59, v29\n\t" \
  "s_nop 1\n\t" \
  "s_bitcmp1_b32 s47, 6\n\ts_cselect_b64 s[78:79], s[52:53], s[50:51]\n\t" \
  "s_bitcmp1_b64 s[78:79], s47\n\t" \
  "s_cbranch_scc1 LBACKC0_%=\n\t" \
  "s_sleep 2\n\t" \
  "s_memrealtime s[62:63]\n\t" \
  "s_waitcnt lgkmcnt(0)\n\t" \
  "s_sub_u32 s62, s62, s60\n\t" \
  "s_cmp_lt_u32 s62, %[sDL]\n\t" \
  "s_cbranch_scc1 LSPINC0_%=\n\t" \
  "v_mov_b32 v30, %[cTMO]\n\t" \
  "buffer_store_dword v27, v30, %[srdC], 0 offen sc1\n\t" \
  "s_waitcnt vmcnt(0)\n\t" \
  "s_mov_b32 s59, 0\n\t" \
  "s_branch LBACKC0_%=\n\t" \
  "LSLOWC1_%=:\n\t" \
  "s_memrealtime s[60:61]\n\t" \
  "s_add_u32 s69, s69, 1\n\t" \
  "LSPINC1_%=:\n\t" \
  "buffer_load_dwordx2 v[28:29], %[vFo], %[srdC], 0 offen sc0 sc1\n\t" \
  "s_waitcnt vmcnt(0) lgkmcnt(0)\n\t" \
  "v_cmp_le_u32 s[50:51], s59, v28\n\t" \
  "v_cmp_le_u32 s[52:53], s59, v29\n\t" \
  "s_nop 1\n\t" \
  "s_bitcmp1_b32 s47, 6\n\ts_cselect_b64 s[78:79], s[52:53], s[50:51]\n\t" \
  "s_bitcmp1_b64 s[78:79], s47\n\t" \
  "s_cbranch_scc1 LBACKC1_%=\n\t" \
  "s_sleep 2\n\t" \
  "s_memrealtime s[62:63]\n\t" \
  "s_waitcnt lgkmcnt(0)\n\t" \
  "s_sub_u32 s62, s62, s60\n\t" \
  "s_cmp_lt_u32 s62, %[sDL]\n\t" \
  "s_cbranch_scc1 LSPINC1_%=\n\t" \
  "v_mov_b32 v30, %[cTMO]\n\t" \
  "buffer_store_dword v27, v30, %[srdC], 0 offen sc1\n\t" \
  "s_waitcnt vmcnt(0)\n\t" \
  "s_mov_b32 s59, 0\n\t" \
  "s_branch LBACKC1_%=\n\t" \
  "LSLOWC2_%=:\n\t" \
  "s_memrealtime s[60:61]\n\t" \
  "s_add_u32 s69, s69, 1\n\t" \
  "LSPINC2_%=:\n\t" \
  "buffer_load_dwordx2 v[28:29], %[vFo], %[srdC], 0 offen sc0 sc1\n\t" \
  "s_waitcnt vmcnt(0) lgkmcnt(0)\n\t" \
  "v_cmp_le_u32 s[50:51], s59, v28\n\t" \
  "v_cmp_le_u32 s[52:53], s59, v29\n\t" \
  "s_nop 1\n\t" \
  "s_bitcmp1_b32 s47, 6\n\ts_cselect_b64 s[78:79], s[52:53], s[50:51]\n\t" \
  "s_bitcmp1_b64 s[78:79], s47\n\t" \
  "s_cbranch_scc1 LBACKC2_%=\n\t" \
  "s_sleep 2\n\t" \
  "s_memrealtime s[62:63]\n\t" \
  "s_waitcnt lgkmcnt(0)\n\t" \
  "s_sub_u32 s62, s62, s60\n\t" \
  "s_cmp_lt_u32 s62, %[sDL]\n\t" \
  "s_cbranch_scc1 LSPINC2_%=\n\t" \
  "v_mov_b32 v30, %[cTMO]\n\t" \
  "buffer_store_dword v27, v30, %[srdC], 0 offen sc1\n\t" \
  "s_waitcnt vmcnt(0)\n\t" \
  "s_mov_b32 s59, 0\n\t" \
  "s_branch LBACKC2_%=\n\t" \
  "LSLOWC3_%=:\n\t" \
  "s_memrealtime s[60:61]\n\t" \
  "s_add_u32 s69, s69, 1\n\t" \
  "LSPINC3_%=:\n\t" \
  "buffer_load_dwordx2 v[28:29], %[vFo], %[srdC], 0 offen sc0 sc1\n\t" \
  "s_waitcnt vmcnt(0) lgkmcnt(0)\n\t" \
  "v_cmp_le_u32 s[50:51], s59, v28\n\t" \
  "v_cmp_le_u32 s[52:53], s59, v29\n\t" \
  "s_nop 1\n\t" \
  "s_bitcmp1_b32 s47, 6\n\ts_cselect_b64 s[78:79], s[52:53], s[50:51]\n\t" \
  "s_bitcmp1_b64 s[78:79], s47\n\t" \
  "s_cbranch_scc1 LBACKC3_%=\n\t" \
  "s_sleep 2\n\t" \
  "s_memrealtime s[62:63]\n\t" \
  "s_waitcnt lgkmcnt(0)\n\t" \
  "s_sub_u32 s62, s62, s60\n\t" \
  "s_cmp_lt_u32 s62, %[sDL]\n\t" \
  "s_cbranch_scc1 LSPINC3_%=\n\t" \
  "v_mov_b32 v30, %[cTMO]\n\t" \
  "buffer_store_dword v27, v30, %[srdC], 0 offen sc1\n\t" \
  "s_waitcnt vmcnt(0)\n\t" \
  "s_mov_b32 s59, 0\n\t" \
  "s_branch LBACKC3_%=\n\t" \
  "LSLOWP0_%=:\n\t" \
  "s_memrealtime s[60:61]\n\t" \
  "s_add_u32 s69, s69, 1\n\t" \
  "LSPINP0_%=:\n\t" \
  "buffer_load_dwordx2 v[28:29], %[vFo], %[srdC], 0 offen sc0 sc1\n\t" \
  "s_waitcnt vmcnt(0) lgkmcnt(0)\n\t" \
  "v_cmp_le_u32 s[50:51], s59, v28\n\t" \
  "v_cmp_le_u32 s[52:53], s59, v29\n\t" \
  "s_nop 1\n\t" \
  "s_bitcmp1_b32 s47, 6\n\ts_cselect_b64 s[78:79], s[52:53], s[50:51]\n\t" \
  "s_bitcmp1_b64 s[78:79], s47\n\t" \
  "s_cbranch_scc1 LBACKP0_%=\n\t" \
  "s_sleep 2\n\t" \
  "s_memrealtime s[62:63]\n\t" \
  "s_waitcnt lgkmcnt(0)\n\t" \
  "s_sub_u32 s62, s62, s60\n\t" \
  "s_cmp_lt_u32 s62, %[sDL]\n\t" \
  "s_cbranch_scc1 LSPINP0_%=\n\t" \
  "v_mov_b32 v30, %[cTMO]\n\t" \
  "buffer_store_dword v27, v30, %[srdC], 0 offen sc1\n\t" \
  "s_waitcnt vmcnt(0)\n\t" \
  "s_mov_b32 s59, 0\n\t" \
  "s_branch LBACKP0_%=\n\t" \
  "LSLOWP1_%=:\n\t" \
  "s_memrealtime s[60:61]\n\t" \
  "s_add_u32 s69, s69, 1\n\t" \
  "LSPINP1_%=:\n\t" \
  "buffer_load_dwordx2 v[28:29], %[vFo], %[srdC], 0 offen sc0 sc1\n\t" \
  "s_waitcnt vmcnt(0) lgkmcnt(0)\n\t" \
  "v_cmp_le_u32 s[50:51], s59, v28\n\t" \
  "v_cmp_le_u32 s[52:53], s59, v29\n\t" \
  "s_nop 1\n\t" \
  "s_bitcmp1_b32 s47, 6\n\ts_cselect_b64 s[78:79], s[52:53], s[50:51]\n\t" \
  "s_bitcmp1_b64 s[78:79], s47\n\t" \
  "s_cbranch_scc1 LBACKP1_%=\n\t" \
  "s_sleep 2\n\t" \
  "s_memrealtime s[62:63]\n\t" \
  "s_waitcnt lgkmcnt(0)\n\t" \
  "s_sub_u32 s62, s62, s60\n\t" \
  "s_cmp_lt_u32 s62, %[sDL]\n\t" \
  "s_cbranch_scc1 LSPINP1_%=\n\t" \
  "v_mov_b32 v30, %[cTMO]\n\t" \
  "buffer_store_dword v27, v30, %[srdC], 0 offen sc1\n\t" \
  "s_waitcnt vmcnt(0)\n\t" \
  "s_mov_b32 s59, 0\n\t" \
  "s_branch LBACKP1_%=\n\t" \
  "LSLOWP2_%=:\n\t" \
  "s_memrealtime s[60:61]\n\t" \
  "s_add_u32 s69, s69, 1\n\t" \
  "LSPINP2_%=:\n\t" \
  "buffer_load_dwordx2 v[28:29], %[vFo], %[srdC], 0 offen sc0 sc1\n\t" \
  "s_waitcnt vmcnt(0) lgkmcnt(0)\n\t" \
  "v_cmp_le_u32 s[50:51], s59, v28\n\t" \
  "v_cmp_le_u32 s[52:53], s59, v29\n\t" \
  "s_nop 1\n\t" \
  "s_bitcmp1_b32 s47, 6\n\ts_cselect_b64 s[78:79], s[52:53], s[50:51]\n\t" \
  "s_bitcmp1_b64 s[78:79], s47\n\t" \
  "s_cbranch_scc1 LBACKP2_%=\n\t" \
  "s_sleep 2\n\t" \
  "s_memrealtime s[62:63]\n\t" \
  "s_waitcnt lgkmcnt(0)\n\t" \
  "s_sub_u32 s62, s62, s60\n\t" \
  "s_cmp_lt_u32 s62, %[sDL]\n\t" \
  "s_cbranch_scc1 LSPINP2_%=\n\t" \
  "v_mov_b32 v30, %[cTMO]\n\t" \
  "buffer_store_dword v27, v30, %[srdC], 0 offen sc1\n\t" \
  "s_waitcnt vmcnt(0)\n\t" \
  "s_mov_b32 s59, 0\n\t" \
  "s_branch LBACKP2_%=\n\t" \
  "LSLOWP3_%=:\n\t" \
  "s_memrealtime s[60:61]\n\t" \
  "s_add_u32 s69, s69, 1\n\t" \
  "LSPINP3_%=:\n\t" \
  "buffer_load_dwordx2 v[28:29], %[vFo], %[srdC], 0 offen sc0 sc1\n\t" \
  "s_waitcnt vmcnt(0) lgkmcnt(0)\n\t" \
  "v_cmp_le_u32 s[50:51], s59, v28\n\t" \
  "v_cmp_le_u32 s[52:53], s59, v29\n\t" \
  "s_nop 1\n\t" \
  "s_bitcmp1_b32 s47, 6\n\ts_cselect_b64 s[78:79], s[52:53], s[50:51]\n\t" \
  "s_bitcmp1_b64 s[78:79], s47\n\t" \
  "s_cbranch_scc1 LBACKP3_%=\n\t" \
  "s_sleep 2\n\t" \
  "s_memrealtime s[62:63]\n\t" \
  "s_waitcnt lgkmcnt(0)\n\t" \
  "s_sub_u32 s62, s62, s60\n\t" \
  "s_cmp_lt_u32 s62, %[sDL]\n\t" \
  "s_cbranch_scc1 LSPINP3_%=\n\t" \
  "v_mov_b32 v30, %[cTMO]\n\t" \
  "buffer_store_dword v27, v30, %[srdC], 0 offen sc1\n\t" \
  "s_waitcnt vmcnt(0)\n\t" \
  "s_mov_b32 s59, 0\n\t" \
  "s_branch LBACKP3_%=\n\t" \
  "LSLOWP4_%=:\n\t" \
  "s_memrealtime s[60:61]\n\t" \
  "s_add_u32 s69, s69, 1\n\t" \
  "LSPINP4_%=:\n\t" \
  "buffer_load_dwordx2 v[28:29], %[vFo], %[srdC], 0 offen sc0 sc1\n\t" \
  "s_waitcnt vmcnt(0) lgkmcnt(0)\n\t" \
  "v_cmp_le_u32 s[50:51], s59, v28\n\t" \
  "v_cmp_le_u32 s[52:53], s59, v29\n\t" \
  "s_nop 1\n\t" \
  "s_bitcmp1_b32 s47, 6\n\ts_cselect_b64 s[78:79], s[52:53], s[50:51]\n\t" \
  "s_bitcmp1_b64 s[78:79], s47\n\t" \
  "s_cbranch_scc1 LBACKP4_%=\n\t" \
  "s_sleep 2\n\t" \
  "s_memrealtime s[62:63]\n\t" \
  "s_waitcnt lgkmcnt(0)\n\t" \
  "s_sub_u32 s62, s62, s60\n\t" \
  "s_cmp_lt_u32 s62, %[sDL]\n\t" \
  "s_cbranch_scc1 LSPINP4_%=\n\t" \
  "v_mov_b32 v30, %[cTMO]\n\t" \
  "buffer_store_dword v27, v30, %[srdC], 0 offen sc1\n\t" \
  "s_waitcnt vmcnt(0)\n\t" \
  "s_mov_b32 s59, 0\n\t" \
  "s_branch LBACKP4_%=\n\t" \
  "LEND_%=:\n\t" \
  "s_waitcnt vmcnt(0) lgkmcnt(0)\n\t" \
  "v_mov_b32 %[oCall], v27\n\t" \
  "s_nop 15\n\t" \
  ""

typedef float f32x4 __attribute__((ext_vector_type(4)));
typedef unsigned u32x4 __attribute__((ext_vector_type(4)));
typedef _Float16 f16x2 __attribute__((ext_vector_type(2)));
typedef float f32x2 __attribute__((ext_vector_type(2)));

#define A8(b) "a" #b "0","a" #b "1","a" #b "2","a" #b "3","a" #b "4","a" #b "5","a" #b "6","a" #b "7","a" #b "8","a" #b "9"
#define V8(b) "v" #b "0","v" #b "1","v" #b "2","v" #b "3","v" #b "4","v" #b "5","v" #b "6","v" #b "7","v" #b "8","v" #b "9"
#define ACLOB "a0","a1","a2","a3","a4","a5","a6","a7","a8","a9",A8(1),A8(2),A8(3),A8(4),A8(5),A8(6),A8(7),A8(8),A8(9),A8(10),A8(11),A8(12),A8(13),A8(14),A8(15),A8(16),A8(17),A8(18),A8(19),A8(20),A8(21),A8(22),A8(23),A8(24),"a250","a251","a252","a253","a254","a255"
#define VCLOB "v16","v17","v18","v19",V8(2),V8(3),V8(4),V8(5),V8(6),V8(7),V8(8),V8(9),V8(10),V8(11),V8(12),V8(13),V8(14),V8(15),V8(16),V8(17),V8(18),V8(19),V8(20),V8(21),V8(22),"v230","v231"
#define SCLOB "s40","s41","s42","s43","s44","s45","s46","s47","s48","s49","s50","s51","s52","s53","s54","s55","s56","s57","s58","s59"

constexpr size_t WS_CNT = 0;
constexpr size_t WS_TMO = 64 << 10;
constexpr size_t WS_AFLAG = 128 << 10;
constexpr size_t WS_PFLAG = 256 << 10;
constexpr size_t WS_A = 1 << 20;
constexpr size_t WS_PART = WS_A + (32ull << 20);
constexpr size_t WS_NEED = WS_PART + (32ull << 20);
constexpr unsigned long long DEADLINE = 50000000ull;

struct P { const float* x; const float* w; float* out; char* ws[8]; int rank; int pad; };

template <int I> __device__ __forceinline__ float rdacc() { float r; asm volatile("v_accvgpr_read_b32 %0, a[%c1]" : "=v"(r) : "i"(I)); return r; }
__device__ __forceinline__ float gelu(float y) { const float k1 = -2.f * 0.7978845608028654f * 1.4426950408889634f, k2 = k1 * 0.044715f;
  float u = y * (k1 + k2 * y * y); return y * __builtin_amdgcn_rcpf(1.f + __builtin_amdgcn_exp2f(u)); }
template <int M, int R> __device__ __forceinline__ f32x4 rd_row() { return f32x4{rdacc<16 * M + R>(), rdacc<16 * M + 4 + R>(), rdacc<16 * M + 8 + R>(), rdacc<16 * M + 12 + R>()}; }
template <int M0, int I> __device__ __forceinline__ void send_all(__amdgpu_buffer_rsrc_t rs, unsigned vo) {
  __builtin_amdgcn_raw_buffer_store_b128(__builtin_bit_cast(u32x4, rd_row<M0 + I / 4, I % 4>()), rs, vo + I * 1024, 0, 16);
  if constexpr (I < 31) send_all<M0, I + 1>(rs, vo); }
template <int M0, int I> __device__ __forceinline__ void fin_all(const u32x4* pv, float* o, bool bad) {
  f32x4 v = rd_row<M0 + I / 4, I % 4>() + __builtin_bit_cast(f32x4, pv[I]);
  v = f32x4{gelu(v[0]), gelu(v[1]), gelu(v[2]), gelu(v[3])};
  if (bad) { const float q = __builtin_nanf(""); v = f32x4{q, q, q, q}; }
  *(f32x4*)(o + (size_t)(16 * (I / 4) + I % 4) * 8192) = v;
  if constexpr (I < 31) fin_all<M0, I + 1>(pv, o, bad); }

template <int H> __device__ __forceinline__ void epilogue(const P& p, int mt, int nt, unsigned wv, unsigned lane, unsigned call) {
  char* ws = p.ws[p.rank];
  unsigned* tmo = (unsigned*)(ws + WS_TMO);
  const unsigned c = lane & 15, g = lane >> 4; const int tile = mt * 32 + nt;
  constexpr int MK = H ? 8 : 0, MS = H ? 0 : 8;
  float* o = p.out + (size_t)(mt * 256 + MK * 16 + 4 * g) * 8192 + nt * 256 + wv * 64 + 4 * c;
  u32x4 pv[32];
  __amdgpu_buffer_rsrc_t rs = __builtin_amdgcn_make_buffer_rsrc((void*)(ws + WS_PART), (short)0, 32 << 20, 0x00020000);
  unsigned vs = (tile * 2 + (1 - H)) * 131072 + wv * 32768 + lane * 16, vk = (tile * 2 + H) * 131072 + wv * 32768 + lane * 16;
  send_all<MS, 0>(rs, vs);
  asm volatile("s_waitcnt vmcnt(0)" ::: "memory");
  unsigned* myf = (unsigned*)(ws + WS_PFLAG) + ((tile * 2 + H) * 4 + wv) * 32; unsigned* pf = (unsigned*)(ws + WS_PFLAG) + ((tile * 2 + 1 - H) * 4 + wv) * 32;
  if (lane == 0) __hip_atomic_store(myf, call, __ATOMIC_RELAXED, __HIP_MEMORY_SCOPE_AGENT);
  unsigned long long t0 = __builtin_amdgcn_s_memrealtime();
  while ((unsigned)__builtin_amdgcn_readfirstlane(__hip_atomic_load(pf, __ATOMIC_RELAXED, __HIP_MEMORY_SCOPE_AGENT)) < call) {
    if (__builtin_amdgcn_s_memrealtime() - t0 > DEADLINE) { if (lane == 0) __hip_atomic_store(tmo, call, __ATOMIC_RELAXED, __HIP_MEMORY_SCOPE_AGENT); break; }
    __builtin_amdgcn_s_sleep(1); }
  asm volatile("s_waitcnt vmcnt(0)" ::: "memory");
  bool bad = (unsigned)__builtin_amdgcn_readfirstlane(__hip_atomic_load(tmo, __ATOMIC_RELAXED, __HIP_MEMORY_SCOPE_AGENT)) == call;
#pragma unroll
  for (int i = 0; i < 32; ++i) pv[i] = __builtin_amdgcn_raw_buffer_load_b128(rs, vk + i * 1024, 0, 16);
  fin_all<MK, 0>(pv, o, bad);
}

#define PN 3
#define PD 2
#define N0 2
#define SCLOB2 "s60","s61","s62","s63","s64","s65","s66","s67","s68","s69","s70","s71","s72","s73","s74","s75","s76","s77","s78","s79","s80","s81","s82","s83"
__device__ __forceinline__ unsigned rfl(unsigned x) { return __builtin_amdgcn_readfirstlane(x); }
__device__ __forceinline__ unsigned aflag_off(unsigned src, unsigned lks, unsigned fmq) { const unsigned l = lks & 15; return (unsigned)WS_AFLAG + ((fmq * 2 + (lks >> 4)) * 128 + (src * 8 + (l & 7)) * 2 + (l >> 3)) * 4; }

__global__ __launch_bounds__(256, 1) void a2a_gemm(P p) {
  extern __shared__ __attribute__((aligned(16))) char lds[];
  const unsigned b = blockIdx.x, xcd = b & 7, j = b >> 3;
  const unsigned mt = j & 3, h = xcd & 1, nt = (xcd >> 1) * 8 + (j >> 2), ph = nt & 3, rank = p.rank;
  const unsigned tid = threadIdx.x, lane = tid & 63, wv = rfl(tid >> 6);
  const unsigned c = lane & 15, g = lane >> 4;
  char* ws = p.ws[rank];
  auto ent = [&](unsigned s) { if (s >= 128) s = 127; const unsigned n = (128 - ph) & ~3u; if (s >= ph && s < ph + n) { const unsigned t = s - ph; s = ph + (t & ~3u) + ((t + mt) & 3); }
    unsigned src, lev; if (s < 16) { src = rank; lev = s; } else { const unsigned q = s - 16; lev = q / 7; src = (rank + 1 + q % 7) & 7; }
    return ((src * 32 + 16 * h + lev) << 7) | ((lev >> 3) << 6) | (src * 8 + (lev & 7)); };
  auto pk = [&](unsigned s) { return ent(s) | (ent(s + 1) << 16); };
  const unsigned perm = ph * 64 + (j >> 4) * 32 + (j & 3) * 8 + xcd, cls = (b >> 1) % PN;
  auto piece_of_slot = [&](int z) -> int { if (z < 0) return -1; if (z < N0) return z; for (int n = N0; n < 14; ++n) if (N0 + ((n - N0) * PN + (int)cls) / PD == z) return n; return -1; };
  auto decode = [&](int n, unsigned& d, unsigned& lks, unsigned& fmq) { const unsigned q = n * 256 + perm, lev = q / 224, idx = q % 224, rest = idx / 7; d = (rank + 1 + idx % 7) & 7; lks = 16 * (rest & 1) + lev; fmq = rest >> 1; };
  unsigned long long vD = 0, vG = 0; unsigned vXo = 0, d, lks, fmq;
  { int n = piece_of_slot((int)lane); if (n >= 0) { decode(n, d, lks, fmq); vD = (unsigned long long)(p.ws[d & 7] + WS_A + (size_t)(rank * 32 + lks) * 65536 + fmq * 4096); }
    n = piece_of_slot((int)lane - 1); if (n >= 0 && wv == 0) { decode(n, d, lks, fmq); vG = (unsigned long long)(p.ws[d & 7] + aflag_off(rank, lks, fmq)); }
    n = piece_of_slot((int)lane + 1); if (n >= 0) { decode(n, d, lks, fmq); vXo = (d * 1024 + fmq * 64) * 4096 + lks * 128; } }
  decode(0, d, lks, fmq); const unsigned sXP = (d * 1024 + fmq * 64) * 4096 + lks * 128;
  const unsigned zn = N0 + ((13 - N0) * PN + PN - 1) / PD + 4;
  const unsigned lf = b >> 4, ll = 2 * (b & 15);
  const unsigned pr = lane >> 2, pg = (lane & 3) ^ (2 * (pr >> 3));
  const unsigned vXa = (wv * 16 + pr) * 4096 + pg * 16, vSo = wv * 1024 + lane * 16;
  const unsigned vLF = lane < 2 ? aflag_off(rank, ll + lane, lf) : (unsigned)WS_CNT + b * 128;
  const unsigned myq = mt * 4 + wv;
  const unsigned vFo = (unsigned)WS_AFLAG + (myq * 2 + h) * 512 + lane * 8;
  const unsigned vW = g * 4 * 32768 + (nt * 256 + wv * 64 + 4 * c) * 4;
  const unsigned vA = lane * 16 + myq * 4096, vDS = (c * 4 + (g ^ (2 * (c >> 3)))) * 16, vTab = pk(2 * lane + 4);
  __amdgpu_buffer_rsrc_t srdW = __builtin_amdgcn_make_buffer_rsrc((void*)p.w, (short)0, 0x10000000, 0x00020000);
  __amdgpu_buffer_rsrc_t srdA = __builtin_amdgcn_make_buffer_rsrc((void*)(ws + WS_A), (short)0, 0x2000000, 0x00020000);
  __amdgpu_buffer_rsrc_t srdX = __builtin_amdgcn_make_buffer_rsrc((void*)p.x, (short)0, 0x2000000, 0x00020000);
  __amdgpu_buffer_rsrc_t srdC = __builtin_amdgcn_make_buffer_rsrc((void*)ws, (short)0, 0x100000, 0x00020000);
  unsigned vcall;
  asm volatile(MAINLOOP : [oCall] "=v"(vcall)
     : [vW] "v"(vW), [vA] "v"(vA), [vDS] "v"(vDS), [vTab] "v"(vTab), [vXa] "v"(vXa), [vSo] "v"(vSo), [vFo] "v"(vFo), [vLF] "v"(vLF),
       [vDlo] "v"((unsigned)vD), [vDhi] "v"((unsigned)(vD >> 32)), [vGlo] "v"((unsigned)vG), [vGhi] "v"((unsigned)(vG >> 32)), [vXo] "v"(vXo),
       [srdW] "s"(srdW), [srdA] "s"(srdA), [srdX] "s"(srdX), [srdC] "s"(srdC),
       [sM0] "s"(rfl(wv * 4096)), [sK0] "s"(rfl(pk(0))), [sK1] "s"(rfl(pk(2))), [sPH] "s"(rfl(ph)), [sZN] "s"(zn), [sXP] "s"(rfl(sXP)),
       [sXL0] "s"(rfl((rank * 1024 + lf * 64) * 4096 + ll * 128)), [sXL1] "s"(rfl((rank * 1024 + lf * 64) * 4096 + ll * 128 + 128)),
       [sAL0] "s"(rfl((rank * 32 + ll) * 65536 + lf * 4096)), [sAL1] "s"(rfl((rank * 32 + ll + 1) * 65536 + lf * 4096)),
       [sCnt] "s"(rfl((unsigned)WS_CNT + b * 128)), [sLFm] "s"((unsigned long long)(wv == 0 ? 7u : 0u)), [sDL] "s"((unsigned)DEADLINE), [cTMO] "i"((int)WS_TMO)
     : "memory", "scc", "vcc", ACLOB, VCLOB, SCLOB, SCLOB2);
  const unsigned call = rfl(vcall);
  if (h) epilogue<1>(p, mt, nt, wv, lane, call); else epilogue<0>(p, mt, nt, wv, lane, call);
}

extern "C" void kernel_launch(int rank, int nranks, ncclComm_t comm, hipStream_t stream, void* const* d_in, const int* in_sizes, int n_in,
                              void* d_out, int out_size, const int* in_dtypes, int out_dtype, void* const* d_out_all, void* const* d_ws_all, size_t ws_bytes) {
  if (nranks != 8 || ws_bytes < WS_NEED || (unsigned)rank >= 8u) { fprintf(stderr, "unsupported configuration\n"); return; }
  hipError_t e = hipFuncSetAttribute((const void*)a2a_gemm, hipFuncAttributeMaxDynamicSharedMemorySize, 131072);
  if (e != hipSuccess) { fprintf(stderr, "hipFuncSetAttribute: %s\n", hipGetErrorString(e)); return; }
  P p; p.x = (const float*)d_in[0]; p.w = (const float*)d_in[1]; p.out = (float*)d_out; p.rank = rank; p.pad = 0;
  for (int i = 0; i < 8; ++i) p.ws[i] = (char*)d_ws_all[i];
  hipLaunchKernelGGL(a2a_gemm, dim3(256), dim3(256), 131072, stream, p);
}
```
